```python
import numpy as np

kernel_name = "gpu_gemm_ar_m4096_k4096_n16384_e4m3e4m3_dqrelu"


ABSMAX_RTOL = 0.03

COMPUTE_DTYPE = "float32"
EPILOGUE = "dequant_relu"
IN_QUANT = None
OUT_QUANT = None
IN_FP8 = "e4m3e4m3"
IN_DTYPES = {"x": "e4m3", "w_mat": "e4m3",
             "scale_x": "float32", "scale_w": "float32"}
WORLD_SIZE = 8

def _snap_fp8(x, mant_bits, max_abs, min_normal):
    a = np.abs(x.astype(np.float64))
    _m, _e = np.frexp(a)
    q = np.where(a >= min_normal, np.ldexp(1.0, _e - 1 - mant_bits),
                 min_normal * 2.0 ** -mant_bits)
    snapped = np.minimum(np.round(a / q) * q, max_abs)
    return (np.sign(x) * snapped).astype(np.float32)

def setup_inputs():

    x = _snap_fp8(np.random.randn(4096, 4096).astype(np.float32), 3, 448.0, 2.0 ** -6)
    w_mat = _snap_fp8(np.random.randn(4096, 16384).astype(np.float32), 3, 448.0, 2.0 ** -6)
    scale_x = (0.02 * (1.0 + np.random.rand(1))).astype(np.float32)
    scale_w = (0.02 * (1.0 + np.random.rand(1))).astype(np.float32)
    return {"x": x, "w_mat": w_mat, "scale_x": scale_x, "scale_w": scale_w}

def reference(x, w_mat, scale_x, scale_w):
    acc = (x @ w_mat).astype(np.float32)
    y = (acc * (scale_x * scale_w)).astype(np.float32)
    return np.maximum(y, 0.0).astype(np.float32)
```

```cpp
#include <hip/hip_runtime.h>
#include <rccl/rccl.h>
#include <cstdio>

#define HIP_CHECK(expr) do { hipError_t _e = (expr); if (_e != hipSuccess) { \
    fprintf(stderr, "HIP error %s at %s:%d\n", hipGetErrorString(_e), __FILE__, __LINE__); \
    return; } } while (0)

namespace {
#define LAS __attribute__((address_space(3)))
typedef float f32x4 __attribute__((ext_vector_type(4)));
typedef int i32x4 __attribute__((ext_vector_type(4)));
typedef unsigned u32x4 __attribute__((ext_vector_type(4)));
typedef unsigned u32x2 __attribute__((ext_vector_type(2)));

constexpr int NR = 8;
constexpr int M = 4096, N = 16384, K = 4096, KS = K / NR;
constexpr int BM = 256, HTB = 16384, STAGE_BYTES = 8 * HTB, NT = K / 128;
constexpr int LDS_BYTES = STAGE_BYTES + 256;
constexpr size_t HBSTRIDE = (size_t)NT * HTB;
constexpr size_t XS_BYTES = (size_t)M * K;
constexpr size_t WS_BYTES = (size_t)N * K;
constexpr size_t BUF_BYTES = XS_BYTES + WS_BYTES;
constexpr size_t CTL_OFF = 2 * BUF_BYTES;
constexpr int NCOMM = 16;
constexpr int GRID = 256;
constexpr int WPX = (GRID - NCOMM) / 8;
constexpr int NROUND = 5;
constexpr int NSTAGE = 4 * NROUND;
constexpr size_t CTL_FLAGS = 0;
constexpr size_t CTL_CNT = 64 * 1024;
constexpr size_t CTL_TMO = 68 * 1024;
constexpr size_t CTL_BYTES = 72 * 1024;
constexpr unsigned long long DEADLINE_TICKS = 40000000ull;

struct Params {
  const unsigned char* x;
  const unsigned char* w;
  const float* sx; const float* sw;
  float* out;
  char* ws[NR];
  int rank;
};

__host__ __device__ constexpr int round_col0(int r) { return 15 * r; }
__host__ __device__ constexpr int round_ncol(int r) { return r < 4 ? 15 : 4; }

struct Unit { int pm, pn; };
__device__ __forceinline__ bool tile_of(int g, int j, int r, Unit& u) {
  const int band = g & 3, h = g >> 2;
  if (r < 4) { const int i = 30 * (2 * r + h) + j; u.pm = band * 4 + (i & 3); u.pn = i >> 2; return true; }
  if (r == 4 && j < 8) { const int i = 8 * h + j; u.pm = band * 4 + (i & 3); u.pn = 60 + (i >> 2); return true; }
  return false;
}

__device__ __forceinline__ unsigned perm(unsigned a, unsigned b, unsigned sel) { return __builtin_amdgcn_perm(a, b, sel); }

__global__ __launch_bounds__(512, 2) void fused(Params P) {
  extern __shared__ __attribute__((aligned(16))) unsigned char shm[];
  LAS unsigned char* lds = (LAS unsigned char*)shm;
  const int tid = threadIdx.x, wid = __builtin_amdgcn_readfirstlane(tid >> 6), lane = tid & 63;
  char* const myws = P.ws[P.rank];
  char* const ctl = myws + CTL_OFF;
  unsigned call;
  {
    unsigned* cnt = (unsigned*)(ctl + CTL_CNT) + blockIdx.x;
    LAS unsigned* l = (LAS unsigned*)(lds + STAGE_BYTES);
    if (tid == 0) { unsigned c = __hip_atomic_load(cnt, __ATOMIC_RELAXED, __HIP_MEMORY_SCOPE_AGENT) + 1u; __hip_atomic_store(cnt, c, __ATOMIC_RELAXED, __HIP_MEMORY_SCOPE_AGENT); l[0] = c; }
    __syncthreads();
    call = __builtin_amdgcn_readfirstlane(l[0]);
    __syncthreads();
  }
  const size_t bufoff = (call & 1u) ? BUF_BYTES : 0;
  unsigned* const tmo = (unsigned*)(ctl + CTL_TMO);

  if (blockIdx.x < NCOMM) {
    const int cj = blockIdx.x;
    __amdgpu_buffer_rsrc_t rs[NR];
#pragma unroll
    for (int q = 0; q < NR; ++q) { const int p = (P.rank + 1 + q) & (NR - 1); rs[q] = __builtin_amdgcn_make_buffer_rsrc(P.ws[p] + bufoff, 0, (unsigned)BUF_BYTES, 0x00020000); }
    const int fr = lane & 15, fq = lane >> 4;
    for (int st = 0; st < NSTAGE; ++st) {
      const int r = st >> 2, kq = st & 3;
      const int nx = (r == 0) ? 16 : 0, nu = nx + round_ncol(r);
      const unsigned T = (unsigned)(P.rank * 4 + kq);
      for (int u = cj; u < nu; u += NCOMM) {
        u32x4 v[4]; unsigned off[4];
        if (u < nx) {
          const int pm = u;
#pragma unroll
          for (int i = 0; i < 4; ++i) {
            const int p = tid + 512 * i, hb = p >> 10, g = (p >> 7) & 7, h = (p >> 6) & 1;
            const int row = pm * 256 + hb * 128 + g * 16 + fr;
            v[i] = *(const u32x4*)(P.x + (size_t)row * KS + kq * 128 + h * 64 + fq * 16);
            off[i] = (unsigned)(((pm * 2 + hb) * NT + T) * HTB) + (unsigned)((p & 1023) * 16);
          }
        } else {
          const int pn = round_col0(r) + (u - nx);
          const int wc = wid >> 1, h = wid & 1;
          const unsigned char* src = P.w + (size_t)(kq * 128 + h * 64 + fq * 16) * N + pn * 256 + wc * 64 + fr * 4;
          unsigned d[16];
#pragma unroll
          for (int j = 0; j < 16; ++j) d[j] = *(const unsigned*)(src + (size_t)j * N);
#pragma unroll
          for (int q4 = 0; q4 < 4; ++q4) {
            const unsigned a0 = d[q4 * 4], a1 = d[q4 * 4 + 1], a2 = d[q4 * 4 + 2], a3 = d[q4 * 4 + 3];
            const unsigned t0 = perm(a1, a0, 0x05010400u), t1 = perm(a1, a0, 0x07030602u);
            const unsigned t2 = perm(a3, a2, 0x05010400u), t3 = perm(a3, a2, 0x07030602u);
            v[0][q4] = perm(t2, t0, 0x05040100u);
            v[1][q4] = perm(t2, t0, 0x07060302u);
            v[2][q4] = perm(t3, t1, 0x05040100u);
            v[3][q4] = perm(t3, t1, 0x07060302u);
          }
#pragma unroll
          for (int c = 0; c < 4; ++c) {
            const int hb = c >> 1, n = c & 1;
            off[c] = (unsigned)XS_BYTES + (unsigned)(((pn * 2 + hb) * NT + T) * HTB) + (unsigned)((wc * 2 + n) * 2048 + h * 1024 + lane * 16);
          }
        }
#pragma unroll
        for (int q = 0; q < NR; ++q) {
#pragma unroll
          for (int i = 0; i < 4; ++i) __builtin_amdgcn_raw_buffer_store_b128(v[i], rs[q], off[i], 0, 17);
        }
      }
      asm volatile("s_waitcnt vmcnt(0)" ::: "memory");
      __syncthreads();
      if (tid < NR) {
        unsigned* f = (unsigned*)(P.ws[tid] + CTL_OFF + CTL_FLAGS + ((size_t)st * NR + P.rank) * 128) + cj;
        __hip_atomic_store(f, call, __ATOMIC_RELAXED, __HIP_MEMORY_SCOPE_SYSTEM);
      }
    }
    return;
  }

  const int wr = wid >> 2, wc = wid & 3, fr = lane & 15, fq = lane >> 4;
  const int gi = blockIdx.x - NCOMM, xg = gi & 7, slot = gi >> 3;
  const char* const As = myws + bufoff; const char* const Bs = As + XS_BYTES;
  const unsigned voff = tid * 16;
  const unsigned ldsw = (unsigned)wid * 1024u;
  const int aoff = wr * 4 * 2048 + lane * 16, boff = wc * 2 * 2048 + lane * 16;
  const unsigned ovoff = (unsigned)((wr * 64 + fq * 4) * N + wc * 64 + fr * 4) * 4u;
  int lvl = 0;
  bool bad = false;
#define SA(b, h) (((b) * 2 + (h)) * HTB)
#define SB(b, h) ((4 + (b) * 2 + (h)) * HTB)
#define STAGE(bufoff_, gbase) do { _Pragma("unroll") for (int _i = 0; _i < 2; ++_i) \
        __builtin_amdgcn_global_load_lds((const unsigned*)((const char*)(gbase) + voff + _i * 8192), (LAS unsigned*)(lds + (bufoff_) + ldsw + _i * 8192), 16, 0, 0); } while (0)
#define LDA(b, h, mp) do { _Pragma("unroll") for (int m = 0; m < 2; ++m) _Pragma("unroll") for (int k = 0; k < 2; ++k) At[m][k] = *(const LAS i32x4*)(lds + SA(b, h) + aoff + ((mp) * 2 + m) * 2048 + k * 1024); } while (0)
#define LDB(b, h) do { _Pragma("unroll") for (int n = 0; n < 2; ++n) _Pragma("unroll") for (int k = 0; k < 2; ++k) Bt[h][n][k] = *(const LAS i32x4*)(lds + SB(b, h) + boff + n * 2048 + k * 1024); } while (0)
#define CAT(x) __builtin_shufflevector(x[0], x[1], 0, 1, 2, 3, 4, 5, 6, 7)
#define MMA(ai, mp, ZERO) do { __builtin_amdgcn_s_setprio(1); _Pragma("unroll") for (int m = 0; m < 2; ++m) _Pragma("unroll") for (int h = 0; h < 2; ++h) _Pragma("unroll") for (int n = 0; n < 2; ++n) \
        acc[ai][(mp) * 2 + m][h][n] = __builtin_amdgcn_mfma_scale_f32_16x16x128_f8f6f4(CAT(At[m]), CAT(Bt[h][n]), (ZERO) ? (f32x4){0.f, 0.f, 0.f, 0.f} : acc[ai][(mp) * 2 + m][h][n], 0, 0, 0, 0, 0, 0); __builtin_amdgcn_s_setprio(0); } while (0)
#define ST(ai, mp, s0, s1, sbase) do { _Pragma("unroll") for (int s = (s0); s < (s1); ++s) { const int m = (mp) * 2 + (s >> 2), j = s & 3; \
        f32x4 v = {acc[ai][m][0][0][j], acc[ai][m][0][1][j], acc[ai][m][1][0][j], acc[ai][m][1][1][j]}; v = v * scale; v = __builtin_elementwise_max(v, lo4); \
        __builtin_amdgcn_raw_buffer_store_b128(__builtin_bit_cast(u32x4, v), orsrc, ovoff + ((sbase) + (unsigned)(((ai) * 128 + m * 16 + j) * N) * 4u), 0, 2); } } while (0)
#define WAIT_V(n) asm volatile("s_waitcnt vmcnt(" #n ")" ::: "memory")
#define WAIT_L(n) asm volatile("s_waitcnt lgkmcnt(" #n ")" ::: "memory")
#define BAR __builtin_amdgcn_s_barrier()
#define SCHED __builtin_amdgcn_sched_barrier(0)
#define KOFF(t) ((size_t)((((t) & 7) << 2) | ((t) >> 3)) * HTB)
#define WAIT_STAGE(s) do { if ((s) >= lvl) { \
      LAS unsigned* lw = (LAS unsigned*)(lds + STAGE_BYTES); \
      if (wid == 0) { \
        int l2 = lvl; const unsigned long long t0 = __builtin_amdgcn_s_memrealtime(); bool dead = false; \
        while (l2 <= (s)) { \
          const u32x2 f = __builtin_bit_cast(u32x2, __hip_atomic_load((const unsigned long long*)(ctl + CTL_FLAGS + (size_t)l2 * (NR * 128) + (lane >> 3) * 128 + (lane & 7) * 8), __ATOMIC_RELAXED, __HIP_MEMORY_SCOPE_SYSTEM)); \
          if (__all(f.x >= call && f.y >= call)) { ++l2; continue; } \
          if (__hip_atomic_load(tmo, __ATOMIC_RELAXED, __HIP_MEMORY_SCOPE_AGENT) == call || __builtin_amdgcn_s_memrealtime() - t0 > DEADLINE_TICKS) { \
            if (lane == 0) __hip_atomic_store(tmo, call, __ATOMIC_RELAXED, __HIP_MEMORY_SCOPE_AGENT); \
            dead = true; l2 = NSTAGE; break; } \
          __builtin_amdgcn_s_sleep(2); \
        } \
        __builtin_amdgcn_fence(__ATOMIC_ACQUIRE, ""); \
        if (lane == 0) { lw[0] = (unsigned)l2; lw[1] = dead ? 1u : 0u; } \
        asm volatile("s_waitcnt vmcnt(0) lgkmcnt(0)" ::: "memory"); \
      } \
      asm volatile("" ::: "memory"); BAR; asm volatile("" ::: "memory"); \
      lvl = __builtin_amdgcn_readfirstlane(lw[0]); bad = bad || (__builtin_amdgcn_readfirstlane(lw[1]) != 0u); \
      asm volatile("s_waitcnt lgkmcnt(0)" ::: "memory"); \
    } } while (0)

  Unit cur, nxt; int ui = 0;
  if (!tile_of(xg, slot, 0, cur)) return;
  f32x4 acc[2][4][2][2];
  i32x4 At[2][2], Bt[2][2][2];
  const size_t hstep = HBSTRIDE, tstep = 2 * HBSTRIDE;
  const char* cA = As + (size_t)cur.pm * tstep; const char* cB = Bs + (size_t)cur.pn * tstep;
  const __amdgpu_buffer_rsrc_t orsrc_real = __builtin_amdgcn_make_buffer_rsrc(P.out, 0, (unsigned)((size_t)M * N * 4), 0x00020000);
  __amdgpu_buffer_rsrc_t orsrc = __builtin_amdgcn_make_buffer_rsrc(P.out, 0, 0u, 0x00020000);
  float scale = P.sx[0] * P.sw[0];
  f32x4 lo4 = {0.f, 0.f, 0.f, 0.f};
  WAIT_STAGE(0);
  STAGE(SB(0, 0), cB); STAGE(SB(0, 1), cB + hstep); STAGE(SA(0, 0), cA); STAGE(SA(0, 1), cA + hstep);
  STAGE(SB(1, 0), cB + KOFF(1)); STAGE(SB(1, 1), cB + hstep + KOFF(1)); STAGE(SA(1, 0), cA + KOFF(1));
  if (wr == 1) BAR;
  WAIT_V(6); BAR; BAR;
  unsigned psb = 0;
#pragma unroll
  for (int a = 0; a < 2; ++a)
#pragma unroll
    for (int m = 0; m < 4; ++m)
#pragma unroll
      for (int h = 0; h < 2; ++h)
#pragma unroll
        for (int n = 0; n < 2; ++n) acc[a][m][h][n] = (f32x4){0.f, 0.f, 0.f, 0.f};
#define ITER(FIRST, LAST, a1, a2, b2, a3, b3) do { \
        LDB(0, 0); SCHED; LDB(0, 1); LDA(0, 0, 0); \
      if (FIRST) { ST(0, 0, 7, 8, psb); ST(0, 1, 5, 7, psb); ST(1, 0, 2, 4, psb); ST(1, 1, 0, 1, psb); } \
      WAIT_L(8); BAR; WAIT_L(0); MMA(0, 0, FIRST); BAR; SCHED; \
        LDA(0, 0, 1); STAGE(SB(0, 0), b2); STAGE(SA(1, 1), (a1) + hstep); \
      if (FIRST) { ST(0, 1, 7, 8, psb); ST(1, 0, 4, 7, psb); WAIT_V(24); } else WAIT_V(8); \
      BAR; WAIT_L(0); MMA(0, 1, FIRST); BAR; SCHED; \
        LDA(0, 1, 0); STAGE(SB(0, 1), (b2) + hstep); \
      if (FIRST) { ST(1, 0, 7, 8, psb); ST(1, 1, 1, 5, psb); } \
      BAR; WAIT_L(0); MMA(1, 0, FIRST); BAR; SCHED; \
        LDA(0, 1, 1); STAGE(SA(0, 0), a2); \
      if (FIRST) { ST(1, 1, 5, 8, psb); WAIT_V(26); } else WAIT_V(8); \
      BAR; WAIT_L(0); MMA(1, 1, FIRST); BAR; SCHED; \
        LDB(1, 0); SCHED; LDB(1, 1); LDA(1, 0, 0); \
      WAIT_L(8); BAR; WAIT_L(0); MMA(0, 0, 0); BAR; SCHED; \
        LDA(1, 0, 1); STAGE(SB(1, 0), b3); STAGE(SA(0, 1), (a2) + hstep); \
      if (LAST) { ST(0, 0, 0, 4, sb); WAIT_V(12); } else if (FIRST) WAIT_V(16); else WAIT_V(8); \
      BAR; WAIT_L(0); MMA(0, 1, 0); BAR; SCHED; \
        LDA(1, 1, 0); STAGE(SB(1, 1), (b3) + hstep); \
      if (LAST) { ST(0, 0, 4, 7, sb); ST(0, 1, 0, 2, sb); } \
      BAR; WAIT_L(0); MMA(1, 0, 0); BAR; SCHED; \
        LDA(1, 1, 1); STAGE(SA(1, 0), a3); \
      if (LAST) { ST(0, 1, 2, 5, sb); ST(1, 0, 0, 2, sb); WAIT_V(22); } else WAIT_V(8); \
      BAR; WAIT_L(0); MMA(1, 1, 0); BAR; SCHED; \
    } while (0)
  for (;;) {
    const bool has_next = tile_of(xg, slot, ui + 1, nxt);
    const char* nA = has_next ? As + (size_t)nxt.pm * tstep : cA; const char* nB = has_next ? Bs + (size_t)nxt.pn * tstep : cB;
    const unsigned sb = (unsigned)(cur.pm * BM * N + cur.pn * BM) * 4u;
    ITER(1, 0, cA + KOFF(1), cA + KOFF(2), cB + KOFF(2), cA + KOFF(3), cB + KOFF(3));
    orsrc = orsrc_real;
    for (int t = 2; t < NT - 2; t += 2) {
      if (((t + 2) & 7) == 0) WAIT_STAGE(ui * 4 + ((t + 2) >> 3));
      ITER(0, 0, cA + KOFF(t + 1), cA + KOFF(t + 2), cB + KOFF(t + 2), cA + KOFF(t + 3), cB + KOFF(t + 3));
    }
    if (has_next) WAIT_STAGE((ui + 1) * 4);
    if (bad) { scale = __builtin_nanf(""); lo4 = (f32x4){scale, scale, scale, scale}; }
    ITER(0, 1, cA + KOFF(NT - 1), nA, nB, nA + KOFF(1), nB + KOFF(1));
    psb = sb;
    if (!has_next) break;
    cur = nxt; cA = nA; cB = nB; ++ui;
  }
  ST(0, 0, 7, 8, psb); ST(0, 1, 5, 8, psb); ST(1, 0, 2, 8, psb); ST(1, 1, 0, 8, psb);
  WAIT_V(0);
  if (wr == 0) BAR;
  BAR;
}
}

extern "C" void kernel_launch(int rank, int nranks, ncclComm_t comm,
                              hipStream_t stream,
                              void* const* d_in, const int* in_sizes, int n_in,
                              void* d_out, int out_size,
                              const int* in_dtypes, int out_dtype,
                              void* const* d_out_all,
                              void* const* d_ws_all, size_t ws_bytes) {
  if (nranks != NR || n_in < 4 || ws_bytes < CTL_OFF + CTL_BYTES || (unsigned)rank >= (unsigned)NR) { fprintf(stderr, "unsupported configuration\n"); return; }
  static bool init = false;
  if (!init) {
    HIP_CHECK(hipFuncSetAttribute((const void*)fused, hipFuncAttributeMaxDynamicSharedMemorySize, LDS_BYTES));
    init = true;
  }
  Params P;
  P.x = (const unsigned char*)d_in[0]; P.w = (const unsigned char*)d_in[1];
  P.sx = (const float*)d_in[2]; P.sw = (const float*)d_in[3];
  P.out = (float*)d_out; P.rank = rank;
  for (int p = 0; p < NR; ++p) P.ws[p] = (char*)d_ws_all[p];
  hipLaunchKernelGGL(fused, dim3(GRID), dim3(512), LDS_BYTES, stream, P);
}
```

```cpp
#include <hip/hip_runtime.h>
#include <rccl/rccl.h>
#include <cstdio>

#define HIP_CHECK(expr) do { hipError_t _e = (expr); if (_e != hipSuccess) { \
    fprintf(stderr, "HIP error %s at %s:%d\n", hipGetErrorString(_e), __FILE__, __LINE__); \
    return; } } while (0)

namespace {
#define LAS __attribute__((address_space(3)))
typedef float f32x4 __attribute__((ext_vector_type(4)));
typedef int i32x4 __attribute__((ext_vector_type(4)));
typedef unsigned u32x4 __attribute__((ext_vector_type(4)));
typedef unsigned u32x2 __attribute__((ext_vector_type(2)));

constexpr int NR = 8;
constexpr int M = 4096, N = 16384, K = 4096, KS = K / NR;
constexpr int BM = 256, HTB = 16384, STAGE_BYTES = 8 * HTB, NT = K / 128;
constexpr int LDS_BYTES = STAGE_BYTES + 256;
constexpr size_t HBSTRIDE = (size_t)4 * HTB;
constexpr size_t XS_BYTES = (size_t)M * KS;
constexpr size_t SRC_STRIDE = (size_t)(M + N) * KS;
constexpr size_t BUF_BYTES = NR * SRC_STRIDE;
constexpr size_t CTL_OFF = 2 * BUF_BYTES;
constexpr int NCOMM = 16;
constexpr int GRID = 256;
constexpr int WPX = (GRID - NCOMM) / 8;
constexpr int NROUND = 5;
constexpr int NSTAGE = 4 * NROUND;
constexpr int NUNIT = 320;
constexpr size_t CTL_FLAGS = 0;
constexpr size_t CTL_CNT = 64 * 1024;
constexpr size_t CTL_TMO = 68 * 1024;
constexpr size_t CTL_BYTES = 72 * 1024;
#define STAMP(i) do { } while (0)
#define COMM_Q 6
#define COMM_ROT 0
#define COMM_NDEST 8
#define POLL_SLEEP 8
constexpr unsigned long long DEADLINE_TICKS = 40000000ull;

struct Params {
  const unsigned char* x;
  const unsigned char* w;
  const float* sx; const float* sw;
  float* out;
  char* ws[NR];
  int rank;
};

__host__ __device__ constexpr int round_col0(int r) { return 15 * r; }
__host__ __device__ constexpr int round_ncol(int r) { return r < 4 ? 15 : 4; }

struct Unit { int pm, pn; };
__device__ __forceinline__ bool tile_of(int g, int j, int r, Unit& u) {
  const int band = g & 3, h = g >> 2;
  if (r < 4) { const int i = 30 * (2 * r + h) + j; u.pm = band * 4 + (i & 3); u.pn = i >> 2; return true; }
  if (r == 4 && j < 8) { const int i = 8 * h + j; u.pm = band * 4 + (i & 3); u.pn = 60 + (i >> 2); return true; }
  return false;
}

__device__ __forceinline__ unsigned perm(unsigned a, unsigned b, unsigned sel) { return __builtin_amdgcn_perm(a, b, sel); }

__global__ __launch_bounds__(512, 2) void fused(Params P) {
  extern __shared__ __attribute__((aligned(16))) unsigned char shm[];
  LAS unsigned char* lds = (LAS unsigned char*)shm;
  const int tid = threadIdx.x, wid = __builtin_amdgcn_readfirstlane(tid >> 6), lane = tid & 63;
  char* const myws = P.ws[P.rank];
  STAMP(0);
  char* const ctl = myws + CTL_OFF;
  unsigned call;
  {
    unsigned* cnt = (unsigned*)(ctl + CTL_CNT) + blockIdx.x;
    LAS unsigned* l = (LAS unsigned*)(lds + STAGE_BYTES);
    if (tid == 0) { unsigned c = __hip_atomic_load(cnt, __ATOMIC_RELAXED, __HIP_MEMORY_SCOPE_AGENT) + 1u; __hip_atomic_store(cnt, c, __ATOMIC_RELAXED, __HIP_MEMORY_SCOPE_AGENT); l[0] = c; }
    __syncthreads();
    call = __builtin_amdgcn_readfirstlane(l[0]);
    __syncthreads();
  }
  const size_t bufoff = (call & 1u) ? BUF_BYTES : 0;
  unsigned* const tmo = (unsigned*)(ctl + CTL_TMO);

  if (blockIdx.x < NCOMM) {
    const int cj = blockIdx.x;
    const int rot = COMM_ROT;
    const int fr = lane & 15, fq = lane >> 4;
    auto stage_end = [](int st) { return st < 4 ? 31 * (st + 1) : (st < 16 ? 124 + 15 * (st - 3) : 304 + 4 * (st - 15)); };
    u32x4 d[4];
    struct UD { int kq, isx, pan; };
    auto decode = [](int g) -> UD {
      UD u;
      const int a = g / 31, ai = g - a * 31;
      const int b = (g - 124) / 15, bi = (g - 124) - b * 15;
      const int c = (g - 304) >> 2, ci = (g - 304) & 3;
      const bool r0 = g < 124, r13 = g < 304;
      u.kq = r0 ? a : (r13 ? (b & 3) : c);
      u.isx = r0 && ai < 16;
      u.pan = r0 ? (ai < 16 ? ai : ai - 16) : (r13 ? 15 * (1 + (b >> 2)) + bi : 60 + ci);
      return u; };
    auto load_unit = [&](int g) {
      const UD u = decode(g); const int kq = u.kq;
      if (u.isx) {
#pragma unroll
        for (int i = 0; i < 4; ++i) { const int p = tid + 512 * i, hb = p >> 10, gg = (p >> 7) & 7, h = (p >> 6) & 1;
          const int row = u.pan * 256 + hb * 128 + gg * 16 + fr;
          d[i] = *(const u32x4*)(P.x + (size_t)row * KS + kq * 128 + h * 64 + fq * 16); }
      } else {
        const int pn = u.pan; const int wc = wid >> 1, h = wid & 1;
        const unsigned char* src = P.w + (size_t)(kq * 128 + h * 64 + fq * 16) * N + pn * 256 + wc * 64 + fr * 4;
#pragma unroll
        for (int j = 0; j < 16; ++j) d[j >> 2][j & 3] = *(const unsigned*)(src + (size_t)j * N);
      } };
    int pub = 0;
    auto arrive_publish = [&](int gnext) {
      asm volatile("" ::: "memory"); __builtin_amdgcn_s_barrier(); asm volatile("" ::: "memory");
      int hi = pub; while (hi < NSTAGE && stage_end(hi) <= gnext) ++hi;
      if (wid == 0) {
        for (int st = pub; st < hi; ++st) if (lane < NR) {
          unsigned* f = (unsigned*)(P.ws[lane] + CTL_OFF + CTL_FLAGS + ((size_t)st * NR + P.rank) * 128) + cj;
          __hip_atomic_store(f, call, __ATOMIC_RELAXED, __HIP_MEMORY_SCOPE_SYSTEM); }
      }
      pub = hi; };
    int g = cj; load_unit(g);
    for (int it = 0; g < NUNIT; ++it, g += NCOMM) {
      const UD u = decode(g);
      const unsigned sbase_ = (unsigned)(P.rank * SRC_STRIDE) + (unsigned)(u.kq * HTB);
      u32x4 v[4]; unsigned off[4];
      if (u.isx) {
#pragma unroll
        for (int i = 0; i < 4; ++i) { const int p = tid + 512 * i, hb = p >> 10; v[i] = d[i];
          off[i] = sbase_ + (unsigned)((u.pan * 2 + hb) * HBSTRIDE) + (unsigned)((p & 1023) * 16); }
      } else {
        const int pn = u.pan; const int wc = wid >> 1, h = wid & 1;
#pragma unroll
        for (int q4 = 0; q4 < 4; ++q4) {
          const unsigned a0 = d[q4][0], a1 = d[q4][1], a2 = d[q4][2], a3 = d[q4][3];
          const unsigned t0 = perm(a1, a0, 0x05010400u), t1 = perm(a1, a0, 0x07030602u);
          const unsigned t2 = perm(a3, a2, 0x05010400u), t3 = perm(a3, a2, 0x07030602u);
          v[0][q4] = perm(t2, t0, 0x05040100u); v[1][q4] = perm(t2, t0, 0x07060302u);
          v[2][q4] = perm(t3, t1, 0x05040100u); v[3][q4] = perm(t3, t1, 0x07060302u);
        }
#pragma unroll
        for (int c = 0; c < 4; ++c) { const int hb = c >> 1, n = c & 1;
          off[c] = sbase_ + (unsigned)XS_BYTES + (unsigned)((pn * 2 + hb) * HBSTRIDE) + (unsigned)((wc * 2 + n) * 2048 + h * 1024 + lane * 16); }
      }
      if (g + NCOMM < NUNIT) load_unit(g + NCOMM);
#pragma unroll
      for (int q = 0; q < NR; ++q) {
        const int pd = (P.rank + 1 + q + rot) & (NR - 1);
        const __amdgpu_buffer_rsrc_t rsq = __builtin_amdgcn_make_buffer_rsrc(P.ws[pd] + bufoff, 0, (unsigned)BUF_BYTES, 0x00020000);
#pragma unroll
        for (int i = 0; i < 4; ++i) __builtin_amdgcn_raw_buffer_store_b128(v[i], rsq, off[i], 0, 17);
        if (q == COMM_Q && it > 0) { asm volatile("s_waitcnt vmcnt(%0)" :: "n"(4 * (COMM_Q + 1)) : "memory"); arrive_publish(g); STAMP(it); }
      }
      if (g < 124 && g + NCOMM >= 124) { asm volatile("s_waitcnt vmcnt(0)" ::: "memory"); arrive_publish(g + NCOMM); }
    }
    asm volatile("s_waitcnt vmcnt(0)" ::: "memory");
    arrive_publish(NUNIT + NCOMM);
    STAMP(40);
    return;
  }

  const int wr = wid >> 2, wc = wid & 3, fr = lane & 15, fq = lane >> 4;
  const int gi = blockIdx.x - NCOMM, xg = gi & 7, slot = gi >> 3;
  const char* const As = myws + bufoff; const char* const Bs = As + XS_BYTES;
  const unsigned voff = tid * 16;
  const unsigned ldsw = (unsigned)wid * 1024u;
  const int aoff = wr * 4 * 2048 + lane * 16, boff = wc * 2 * 2048 + lane * 16;
  const unsigned ovoff = (unsigned)((wr * 64 + fq * 4) * N + wc * 64 + fr * 4) * 4u;
  int lvl = 0;
  bool bad = false;
#define SA(b, h) (((b) * 2 + (h)) * HTB)
#define SB(b, h) ((4 + (b) * 2 + (h)) * HTB)
#define STAGE(bufoff_, gbase) do { _Pragma("unroll") for (int _i = 0; _i < 2; ++_i) \
        __builtin_amdgcn_global_load_lds((const unsigned*)((const char*)(gbase) + voff + _i * 8192), (LAS unsigned*)(lds + (bufoff_) + ldsw + _i * 8192), 16, 0, 0); } while (0)
#define LDA(b, h, mp) do { _Pragma("unroll") for (int m = 0; m < 2; ++m) _Pragma("unroll") for (int k = 0; k < 2; ++k) At[m][k] = *(const LAS i32x4*)(lds + SA(b, h) + aoff + ((mp) * 2 + m) * 2048 + k * 1024); } while (0)
#define LDB(b, h) do { _Pragma("unroll") for (int n = 0; n < 2; ++n) _Pragma("unroll") for (int k = 0; k < 2; ++k) Bt[h][n][k] = *(const LAS i32x4*)(lds + SB(b, h) + boff + n * 2048 + k * 1024); } while (0)
#define CAT(x) __builtin_shufflevector(x[0], x[1], 0, 1, 2, 3, 4, 5, 6, 7)
#define MMA(ai, mp, ZERO) do { __builtin_amdgcn_s_setprio(1); _Pragma("unroll") for (int m = 0; m < 2; ++m) _Pragma("unroll") for (int h = 0; h < 2; ++h) _Pragma("unroll") for (int n = 0; n < 2; ++n) { \
        if (ZERO) asm volatile("v_mfma_f32_16x16x128_f8f6f4 %0, %1, %2, 0" : "=v"(acc[ai][(mp) * 2 + m][h][n]) : "v"(CAT(At[m])), "v"(CAT(Bt[h][n]))); \
        else asm volatile("v_mfma_f32_16x16x128_f8f6f4 %0, %1, %2, %0" : "+v"(acc[ai][(mp) * 2 + m][h][n]) : "v"(CAT(At[m])), "v"(CAT(Bt[h][n]))); } \
        __builtin_amdgcn_s_setprio(0); } while (0)
#define ST(ai, mp, s0, s1, sbase) do { _Pragma("unroll") for (int s = (s0); s < (s1); ++s) { const int m = (mp) * 2 + (s >> 2), j = s & 3; \
        f32x4 v = {acc[ai][m][0][0][j], acc[ai][m][0][1][j], acc[ai][m][1][0][j], acc[ai][m][1][1][j]}; v = v * scale; v = __builtin_elementwise_max(v, (f32x4){lo, lo, lo, lo}); \
        __builtin_amdgcn_raw_buffer_store_b128(__builtin_bit_cast(u32x4, v), orsrc, ovoff + ((sbase) + (unsigned)(((ai) * 128 + m * 16 + j) * N) * 4u), 0, 2); } } while (0)
#define WAIT_V(n) asm volatile("s_waitcnt vmcnt(" #n ")" ::: "memory")
#define WAIT_L(n) asm volatile("s_waitcnt lgkmcnt(" #n ")" ::: "memory")
#define BAR __builtin_amdgcn_s_barrier()
#define SCHED __builtin_amdgcn_sched_barrier(0)
#define KOFF(t) ((size_t)((t) & 7) * SRC_STRIDE + (size_t)((t) >> 3) * HTB)
#define WAIT_STAGE(s) do { if ((s) >= lvl) { \
      LAS unsigned* lw = (LAS unsigned*)(lds + STAGE_BYTES); \
      if (wid == 0) { \
        int l2 = lvl; const unsigned long long t0 = __builtin_amdgcn_s_memrealtime(); bool dead = false; \
        const char* fb = ctl + CTL_FLAGS + (lane >> 3) * 128 + (lane & 7) * 8; \
        while (l2 <= (s)) { \
          const unsigned long long f = __hip_atomic_load((const unsigned long long*)(fb + (size_t)l2 * (NR * 128)), __ATOMIC_RELAXED, __HIP_MEMORY_SCOPE_SYSTEM); \
          if (__all((unsigned)f >= call && (unsigned)(f >> 32) >= call)) { ++l2; continue; } \
          if (__hip_atomic_load(tmo, __ATOMIC_RELAXED, __HIP_MEMORY_SCOPE_AGENT) == call || __builtin_amdgcn_s_memrealtime() - t0 > DEADLINE_TICKS) { \
            if (lane == 0) __hip_atomic_store(tmo, call, __ATOMIC_RELAXED, __HIP_MEMORY_SCOPE_AGENT); \
            dead = true; l2 = NSTAGE; break; } \
          __builtin_amdgcn_s_sleep(POLL_SLEEP); \
        } \
        if (l2 < NSTAGE) { \
          unsigned long long f[4]; \
          _Pragma("unroll") for (int q = 0; q < 4; ++q) { const int sq = (l2 + q < NSTAGE) ? l2 + q : NSTAGE - 1; \
            f[q] = __hip_atomic_load((const unsigned long long*)(fb + (size_t)sq * (NR * 128)), __ATOMIC_RELAXED, __HIP_MEMORY_SCOPE_SYSTEM); } \
          bool run = true; \
          _Pragma("unroll") for (int q = 0; q < 4; ++q) { run = run && __all((unsigned)f[q] >= call && (unsigned)(f[q] >> 32) >= call); if (run) ++l2; } \
          if (l2 > NSTAGE) l2 = NSTAGE; \
        } \
        __builtin_amdgcn_fence(__ATOMIC_ACQUIRE, ""); \
        if (lane == 0) { lw[0] = (unsigned)l2; lw[1] = dead ? 1u : 0u; } \
        asm volatile("s_waitcnt vmcnt(0) lgkmcnt(0)" ::: "memory"); \
      } \
      asm volatile("" ::: "memory"); BAR; asm volatile("" ::: "memory"); \
      lvl = __builtin_amdgcn_readfirstlane(lw[0]); bad = bad || (__builtin_amdgcn_readfirstlane(lw[1]) != 0u); \
      asm volatile("s_waitcnt lgkmcnt(0)" ::: "memory"); \
    } } while (0)

  Unit cur, nxt; int ui = 0;
  if (!tile_of(xg, slot, 0, cur)) return;
  f32x4 acc[2][4][2][2];
  i32x4 At[2][2], Bt[2][2][2];
  const size_t hstep = HBSTRIDE, tstep = 2 * HBSTRIDE;
  const char* cA = As + (size_t)cur.pm * tstep; const char* cB = Bs + (size_t)cur.pn * tstep;
  const __amdgpu_buffer_rsrc_t orsrc_real = __builtin_amdgcn_make_buffer_rsrc(P.out, 0, (unsigned)((size_t)M * N * 4), 0x00020000);
  __amdgpu_buffer_rsrc_t orsrc = __builtin_amdgcn_make_buffer_rsrc(P.out, 0, 0u, 0x00020000);
  float scale = __builtin_bit_cast(float, __builtin_amdgcn_readfirstlane(__builtin_bit_cast(int, P.sx[0] * P.sw[0])));
  float lo = 0.f;
  WAIT_STAGE(0);
  STAMP(1);
  STAGE(SB(0, 0), cB); STAGE(SB(0, 1), cB + hstep); STAGE(SA(0, 0), cA); STAGE(SA(0, 1), cA + hstep);
  STAGE(SB(1, 0), cB + KOFF(1)); STAGE(SB(1, 1), cB + hstep + KOFF(1)); STAGE(SA(1, 0), cA + KOFF(1));
  if (wr == 1) BAR;
  WAIT_V(6); BAR; BAR;
  unsigned psb = 0;
#pragma unroll
  for (int a = 0; a < 2; ++a)
#pragma unroll
    for (int m = 0; m < 4; ++m)
#pragma unroll
      for (int h = 0; h < 2; ++h)
#pragma unroll
        for (int n = 0; n < 2; ++n) acc[a][m][h][n] = (f32x4){0.f, 0.f, 0.f, 0.f};
#define ITER(FIRST, LAST, a1, a2, b2, a3, b3) do { \
        LDB(0, 0); SCHED; LDB(0, 1); LDA(0, 0, 0); \
      if (FIRST) { ST(0, 1, 4, 8, psb); } \
      WAIT_L(8); BAR; WAIT_L(0); MMA(0, 0, FIRST); BAR; SCHED; \
        LDA(0, 0, 1); STAGE(SB(0, 0), b2); STAGE(SA(1, 1), (a1) + hstep); \
      if (FIRST) { ST(1, 0, 4, 8, psb); WAIT_V(24); } else WAIT_V(8); \
      BAR; WAIT_L(0); MMA(0, 1, FIRST); BAR; SCHED; \
        LDA(0, 1, 0); STAGE(SB(0, 1), (b2) + hstep); \
      if (FIRST) { ST(1, 1, 0, 4, psb); } \
      BAR; WAIT_L(0); MMA(1, 0, FIRST); BAR; SCHED; \
        LDA(0, 1, 1); STAGE(SA(0, 0), a2); \
      if (FIRST) { ST(1, 1, 4, 8, psb); WAIT_V(24); } else WAIT_V(8); \
      BAR; WAIT_L(0); MMA(1, 1, FIRST); BAR; SCHED; \
        LDB(1, 0); SCHED; LDB(1, 1); LDA(1, 0, 0); \
      WAIT_L(8); BAR; WAIT_L(0); MMA(0, 0, 0); BAR; SCHED; \
        LDA(1, 0, 1); STAGE(SB(1, 0), b3); STAGE(SA(0, 1), (a2) + hstep); \
      if (LAST) { ST(0, 0, 0, 4, sb); WAIT_V(12); } else if (FIRST) WAIT_V(16); else WAIT_V(8); \
      BAR; WAIT_L(0); MMA(0, 1, 0); BAR; SCHED; \
        LDA(1, 1, 0); STAGE(SB(1, 1), (b3) + hstep); \
      if (LAST) { ST(0, 0, 4, 8, sb); } \
      BAR; WAIT_L(0); MMA(1, 0, 0); BAR; SCHED; \
        LDA(1, 1, 1); STAGE(SA(1, 0), a3); \
      if (LAST) { ST(0, 1, 0, 4, sb); ST(1, 0, 0, 4, sb); WAIT_V(24); } else WAIT_V(8); \
      BAR; WAIT_L(0); MMA(1, 1, 0); BAR; SCHED; \
    } while (0)
  for (;;) {
    const bool has_next = tile_of(xg, slot, ui + 1, nxt);
    const char* nA = has_next ? As + (size_t)nxt.pm * tstep : cA; const char* nB = has_next ? Bs + (size_t)nxt.pn * tstep : cB;
    const unsigned sb = (unsigned)(cur.pm * BM * N + cur.pn * BM) * 4u;
    ITER(1, 0, cA + KOFF(1), cA + KOFF(2), cB + KOFF(2), cA + KOFF(3), cB + KOFF(3));
    orsrc = orsrc_real;
    for (int t = 2; t < NT - 2; t += 2) {
      if (((t + 2) & 7) == 0) { WAIT_STAGE(ui * 4 + ((t + 2) >> 3)); }
      ITER(0, 0, cA + KOFF(t + 1), cA + KOFF(t + 2), cB + KOFF(t + 2), cA + KOFF(t + 3), cB + KOFF(t + 3));
    }
    STAMP(2 + ui * 2);
    if (has_next) WAIT_STAGE((ui + 1) * 4);
    STAMP(3 + ui * 2);
    if (bad) { scale = __builtin_nanf(""); lo = scale; }
    ITER(0, 1, cA + KOFF(NT - 1), nA, nB, nA + KOFF(1), nB + KOFF(1));
    psb = sb;
    if (!has_next) break;
    cur = nxt; cA = nA; cB = nB; ++ui;
  }
  ST(0, 1, 4, 8, psb); ST(1, 0, 4, 8, psb); ST(1, 1, 0, 8, psb);
  WAIT_V(0);
  if (wr == 0) BAR;
  BAR;
  STAMP(63);
}
}

extern "C" void kernel_launch(int rank, int nranks, ncclComm_t comm,
                              hipStream_t stream,
                              void* const* d_in, const int* in_sizes, int n_in,
                              void* d_out, int out_size,
                              const int* in_dtypes, int out_dtype,
                              void* const* d_out_all,
                              void* const* d_ws_all, size_t ws_bytes) {
  if (nranks != NR || n_in < 4 || ws_bytes < CTL_OFF + CTL_BYTES || (unsigned)rank >= (unsigned)NR) { fprintf(stderr, "unsupported configuration\n"); return; }
  static bool init = false;
  if (!init) {
    HIP_CHECK(hipFuncSetAttribute((const void*)fused, hipFuncAttributeMaxDynamicSharedMemorySize, LDS_BYTES));
    init = true;
  }
  Params P;
  P.x = (const unsigned char*)d_in[0]; P.w = (const unsigned char*)d_in[1];
  P.sx = (const float*)d_in[2]; P.sw = (const float*)d_in[3];
  P.out = (float*)d_out; P.rank = rank;
  for (int p = 0; p < NR; ++p) P.ws[p] = (char*)d_ws_all[p];
  hipLaunchKernelGGL(fused, dim3(GRID), dim3(512), LDS_BYTES, stream, P);
}
```

```cpp
#include <hip/hip_runtime.h>
#include <rccl/rccl.h>
#include <cstdio>

#define HIP_CHECK(expr) do { hipError_t _e = (expr); if (_e != hipSuccess) { \
    fprintf(stderr, "HIP error %s at %s:%d\n", hipGetErrorString(_e), __FILE__, __LINE__); \
    return; } } while (0)

namespace {
#define LAS __attribute__((address_space(3)))
typedef float f32x4 __attribute__((ext_vector_type(4)));
typedef int i32x4 __attribute__((ext_vector_type(4)));
typedef unsigned u32x4 __attribute__((ext_vector_type(4)));
typedef unsigned u32x2 __attribute__((ext_vector_type(2)));

constexpr int NR = 8;
constexpr int M = 4096, N = 16384, K = 4096, KS = K / NR;
constexpr int BM = 256, HTB = 16384, STAGE_BYTES = 8 * HTB, NT = K / 128;
constexpr int LDS_BYTES = STAGE_BYTES + 256;
constexpr size_t HBSTRIDE = (size_t)4 * HTB;
constexpr size_t XS_BYTES = (size_t)M * KS;
constexpr size_t SRC_STRIDE = (size_t)(M + N) * KS;
constexpr size_t BUF_BYTES = NR * SRC_STRIDE;
constexpr size_t CTL_OFF = 2 * BUF_BYTES;
#define NCOMM_ 16
constexpr int NCOMM = NCOMM_;
constexpr int GRID = 256;
constexpr int WPX = (GRID - NCOMM) / 8;
constexpr int NROUND = 5;
constexpr int NSTAGE = 4 * NROUND;
constexpr int NUNIT = 320;
constexpr size_t CTL_FLAGS = 0;
constexpr size_t CTL_CNT = 64 * 1024;
constexpr size_t CTL_TMO = 68 * 1024;
constexpr size_t CTL_BYTES = 72 * 1024;
#define STAMP(i) do { } while (0)
#define STAMPW(i) do { } while (0)
#define WACC_IN do { } while (0)
#define WACC_OUT do { } while (0)
#define WSTAMP(i) do { } while (0)
#define COMM_HF 1
#define COMM_B 2
#define DRAIN_END 139
#define COMM_AUX 17
#define COMM_Q 6
#define COMM_ROT 0
#define COMM_NDEST 8
#define POLL_SLEEP 8
constexpr unsigned long long DEADLINE_TICKS = 40000000ull;

struct Params {
  const unsigned char* x;
  const unsigned char* w;
  const float* sx; const float* sw;
  float* out;
  char* ws[NR];
  int rank;
};

__host__ __device__ constexpr int round_col0(int r) { return 15 * r; }
__host__ __device__ constexpr int round_ncol(int r) { return r < 4 ? 15 : 4; }

__host__ __device__ constexpr int stage_end(int st) { return st < 4 ? 31 * (st + 1) : (st < 16 ? 124 + 15 * (st - 3) : 304 + 4 * (st - 15)); }
__device__ __forceinline__ int level_of(int B) { return B < 124 ? B / 31 : (B < 304 ? 4 + (B - 124) / 15 : (B >= NUNIT ? NSTAGE : 16 + (B - 304) / 4)); }
struct Unit { int pm, pn; };
__device__ __forceinline__ bool tile_of(int g, int j, int r, Unit& u) {
  const int band = g & 3, h = g >> 2;
  if (r < 4) { const int i = 30 * (2 * r + h) + j; u.pm = band * 4 + (i & 3); u.pn = i >> 2; return true; }
  return false;
}

__device__ __forceinline__ unsigned perm(unsigned a, unsigned b, unsigned sel) { return __builtin_amdgcn_perm(a, b, sel); }

__global__ __launch_bounds__(512, 2) void fused(Params P) {
  extern __shared__ __attribute__((aligned(16))) unsigned char shm[];
  LAS unsigned char* lds = (LAS unsigned char*)shm;
  const int tid = threadIdx.x, wid = __builtin_amdgcn_readfirstlane(tid >> 6), lane = tid & 63;
  char* const myws = P.ws[P.rank];
  STAMP(0);
  char* const ctl = myws + CTL_OFF;
  unsigned call;
  {
    unsigned* cnt = (unsigned*)(ctl + CTL_CNT) + blockIdx.x;
    LAS unsigned* l = (LAS unsigned*)(lds + STAGE_BYTES);
    if (tid == 0) { unsigned c = __hip_atomic_load(cnt, __ATOMIC_RELAXED, __HIP_MEMORY_SCOPE_AGENT) + 1u; __hip_atomic_store(cnt, c, __ATOMIC_RELAXED, __HIP_MEMORY_SCOPE_AGENT); l[0] = c; }
    __syncthreads();
    call = __builtin_amdgcn_readfirstlane(l[0]);
    __syncthreads();
  }
  const size_t bufoff = (call & 1u) ? BUF_BYTES : 0;
  unsigned* const tmo = (unsigned*)(ctl + CTL_TMO);

  if (blockIdx.x < NCOMM) {
    const int cj = blockIdx.x;
    const int rot = COMM_ROT;
    const int fr = lane & 15, fq = lane >> 4;
    u32x4 d[4];
    struct UD { int kq, isx, pan; };
    auto decode = [](int g) -> UD {
      UD u;
      const int a = g / 31, ai = g - a * 31;
      const int b = (g - 124) / 15, bi = (g - 124) - b * 15;
      const int c = (g - 304) >> 2, ci = (g - 304) & 3;
      const bool r0 = g < 124, r13 = g < 304;
      u.kq = r0 ? a : (r13 ? (b & 3) : c);
      u.isx = r0 && ai < 16;
      u.pan = r0 ? (ai < 16 ? ai : ai - 16) : (r13 ? 15 * (1 + (b >> 2)) + bi : 60 + ci);
      return u; };
    auto load_unit = [&](int g) {
      const UD u = decode(g); const int kq = u.kq;
      if (u.isx) {
#pragma unroll
        for (int i = 0; i < 4; ++i) { const int p = tid + 512 * i, hb = p >> 10, gg = (p >> 7) & 7, h = (p >> 6) & 1;
          const int row = u.pan * 256 + hb * 128 + gg * 16 + fr;
          d[i] = *(const u32x4*)(P.x + (size_t)row * KS + kq * 128 + h * 64 + fq * 16); }
      } else {
        const int wc = wid >> 1, h = wid & 1;
        const unsigned char* src = P.w + (size_t)(kq * 128 + h * 64 + fq * 16) * N + u.pan * 256 + wc * 64 + fr * 4;
#pragma unroll
        for (int j = 0; j < 16; ++j) d[j >> 2][j & 3] = *(const unsigned*)(src + (size_t)j * N);
      } };
    auto publish = [&](int n) {
      asm volatile("" ::: "memory"); __builtin_amdgcn_s_barrier(); asm volatile("" ::: "memory");
      if (wid == 0 && lane < NR) {
        unsigned* f = (unsigned*)(P.ws[lane] + CTL_OFF + CTL_FLAGS + (size_t)P.rank * 128) + cj;
        __hip_atomic_store(f, call * 32u + (unsigned)n, __ATOMIC_RELAXED, __HIP_MEMORY_SCOPE_SYSTEM); } };
    bool drained = false;
    int g = cj; load_unit(g);
    for (int it = 0; g < NUNIT; ++it, g += NCOMM) {
      const UD u = decode(g);
      const unsigned sbase_ = (unsigned)(P.rank * SRC_STRIDE) + (unsigned)(u.kq * HTB);
      u32x4 v[4]; unsigned off[4];
      if (u.isx) {
#pragma unroll
        for (int i = 0; i < 4; ++i) { const int p = tid + 512 * i, hb = p >> 10; v[i] = d[i];
          off[i] = sbase_ + (unsigned)((u.pan * 2 + hb) * HBSTRIDE) + (unsigned)((p & 1023) * 16); }
      } else {
        const int wc = wid >> 1, h = wid & 1;
#pragma unroll
        for (int q4 = 0; q4 < 4; ++q4) {
          const unsigned a0 = d[q4][0], a1 = d[q4][1], a2 = d[q4][2], a3 = d[q4][3];
          const unsigned t0 = perm(a1, a0, 0x05010400u), t1 = perm(a1, a0, 0x07030602u);
          const unsigned t2 = perm(a3, a2, 0x05010400u), t3 = perm(a3, a2, 0x07030602u);
          v[0][q4] = perm(t2, t0, 0x05040100u); v[1][q4] = perm(t2, t0, 0x07060302u);
          v[2][q4] = perm(t3, t1, 0x05040100u); v[3][q4] = perm(t3, t1, 0x07060302u);
        }
#pragma unroll
        for (int c = 0; c < 4; ++c) { const int hb = c >> 1, n = c & 1;
          off[c] = sbase_ + (unsigned)XS_BYTES + (unsigned)((u.pan * 2 + hb) * HBSTRIDE) + (unsigned)((wc * 2 + n) * 2048 + h * 1024 + lane * 16); }
      }
      if (g + NCOMM < NUNIT) load_unit(g + NCOMM);
#pragma unroll
      for (int hf = 0; hf < 2; ++hf) {
#pragma unroll
        for (int q = 0; q < NR; ++q) {
          const int pd = (P.rank + 1 + q + rot) & (NR - 1);
          const __amdgpu_buffer_rsrc_t rsq = __builtin_amdgcn_make_buffer_rsrc(P.ws[pd] + bufoff, 0, (unsigned)BUF_BYTES, 0x00020000);
#pragma unroll
          for (int i = 0; i < 2; ++i) __builtin_amdgcn_raw_buffer_store_b128(v[hf * 2 + i], rsq, off[hf * 2 + i], 0, COMM_AUX);
          if (hf == COMM_HF && q == 6 && it > 0 && !drained) { asm volatile("s_waitcnt vmcnt(%0)" :: "n"(16 * COMM_HF + 14) : "memory"); publish(it); STAMP(it); }
        }
      }
      drained = false;
      if (g >= 124 - NCOMM && g < DRAIN_END) { asm volatile("s_waitcnt vmcnt(0)" ::: "memory"); publish(it + 1); drained = true; }
    }
    asm volatile("s_waitcnt vmcnt(0)" ::: "memory");
    publish(NUNIT / NCOMM);
    STAMP(40);
  }

  const int wr = wid >> 2, wc = wid & 3, fr = lane & 15, fq = lane >> 4;
  const int gi = blockIdx.x - NCOMM, xg = gi & 7, slot = gi >> 3;
  const char* const As = myws + bufoff; const char* const Bs = As + XS_BYTES;
  const unsigned voff = tid * 16;
  const unsigned ldsw = (unsigned)wid * 1024u;
  const int aoff = wr * 4 * 2048 + lane * 16, boff = wc * 2 * 2048 + lane * 16;
  const unsigned ovoff = (unsigned)((wr * 64 + fq * 4) * N + wc * 64 + fr * 4) * 4u;
  int lvl = 0;
  bool bad = false;
#define SA(b, h) (((b) * 2 + (h)) * HTB)
#define SB(b, h) ((4 + (b) * 2 + (h)) * HTB)
#define STAGE(bufoff_, gbase) do { _Pragma("unroll") for (int _i = 0; _i < 2; ++_i) \
        __builtin_amdgcn_global_load_lds((const unsigned*)((const char*)(gbase) + voff + _i * 8192), (LAS unsigned*)(lds + (bufoff_) + ldsw + _i * 8192), 16, 0, 0); } while (0)
#define LDA(b, h, mp) do { _Pragma("unroll") for (int m = 0; m < 2; ++m) _Pragma("unroll") for (int k = 0; k < 2; ++k) At[m][k] = *(const LAS i32x4*)(lds + SA(b, h) + aoff + ((mp) * 2 + m) * 2048 + k * 1024); } while (0)
#define LDB(b, h) do { _Pragma("unroll") for (int n = 0; n < 2; ++n) _Pragma("unroll") for (int k = 0; k < 2; ++k) Bt[h][n][k] = *(const LAS i32x4*)(lds + SB(b, h) + boff + n * 2048 + k * 1024); } while (0)
#define CAT(x) __builtin_shufflevector(x[0], x[1], 0, 1, 2, 3, 4, 5, 6, 7)
#define MMA(ai, mp, ZERO) do { __builtin_amdgcn_s_setprio(1); _Pragma("unroll") for (int m = 0; m < 2; ++m) _Pragma("unroll") for (int h = 0; h < 2; ++h) _Pragma("unroll") for (int n = 0; n < 2; ++n) { \
        if (ZERO) asm volatile("v_mfma_f32_16x16x128_f8f6f4 %0, %1, %2, 0" : "=v"(acc[ai][(mp) * 2 + m][h][n]) : "v"(CAT(At[m])), "v"(CAT(Bt[h][n]))); \
        else asm volatile("v_mfma_f32_16x16x128_f8f6f4 %0, %1, %2, %0" : "+v"(acc[ai][(mp) * 2 + m][h][n]) : "v"(CAT(At[m])), "v"(CAT(Bt[h][n]))); } \
        __builtin_amdgcn_s_setprio(0); } while (0)
#define ST(ai, mp, s0, s1, sbase) do { _Pragma("unroll") for (int s = (s0); s < (s1); ++s) { const int m = (mp) * 2 + (s >> 2), j = s & 3; \
        f32x4 v = {acc[ai][m][0][0][j], acc[ai][m][0][1][j], acc[ai][m][1][0][j], acc[ai][m][1][1][j]}; v = v * scale; v = __builtin_elementwise_max(v, (f32x4){lo, lo, lo, lo}); \
        __builtin_amdgcn_raw_buffer_store_b128(__builtin_bit_cast(u32x4, v), orsrc, ovoff + ((sbase) + (unsigned)(((ai) * 128 + m * 16 + j) * N) * 4u), 0, 2); } } while (0)
#define WAIT_V(n) asm volatile("s_waitcnt vmcnt(" #n ")" ::: "memory")
#define WAIT_L(n) asm volatile("s_waitcnt lgkmcnt(" #n ")" ::: "memory")
#define BAR __builtin_amdgcn_s_barrier()
#define SCHED __builtin_amdgcn_sched_barrier(0)
#define KOFF(t) ((size_t)((t) & 7) * SRC_STRIDE + (size_t)((t) >> 3) * HTB)
  auto level_from = [&](unsigned long long f0) -> int {
    const unsigned base = call * 32u; int B = 1 << 20;
#pragma unroll
    for (int k = 0; k < 2; ++k) {
      const unsigned v0 = (unsigned)(f0 >> (32 * k));
      const int n = (v0 > base && v0 - base < 32u) ? (int)(v0 - base) : 0;
      const int b = (n >= NUNIT / NCOMM) ? (1 << 20) : (lane & 7) * 2 + k + NCOMM * n;
      B = min(B, b);
    }
#pragma unroll
    for (int o = 32; o >= 1; o >>= 1) B = min(B, __shfl_xor(B, o));
    return level_of(__builtin_amdgcn_readfirstlane(B)); };
  const char* const fb = ctl + CTL_FLAGS + (lane >> 3) * 128 + (lane & 7) * 8;
  auto probe_level = [&]() -> int { return level_from(__hip_atomic_load((const unsigned long long*)fb, __ATOMIC_RELAXED, __HIP_MEMORY_SCOPE_SYSTEM)); };
#define WAIT_STAGE(s) do { if ((s) >= lvl) { \
      LAS unsigned* lw = (LAS unsigned*)(lds + STAGE_BYTES); \
      WACC_IN; \
      if (wid == 0) { \
        int l2; const unsigned long long t0 = __builtin_amdgcn_s_memrealtime(); bool dead = false; \
        for (;;) { \
          l2 = probe_level(); \
          if (l2 > (s)) break; \
          if (__hip_atomic_load(tmo, __ATOMIC_RELAXED, __HIP_MEMORY_SCOPE_AGENT) == call || __builtin_amdgcn_s_memrealtime() - t0 > DEADLINE_TICKS) { \
            if (lane == 0) __hip_atomic_store(tmo, call, __ATOMIC_RELAXED, __HIP_MEMORY_SCOPE_AGENT); \
            dead = true; l2 = NSTAGE; break; } \
          __builtin_amdgcn_s_sleep(POLL_SLEEP); \
        } \
        __builtin_amdgcn_fence(__ATOMIC_ACQUIRE, ""); \
        if (lane == 0) { lw[0] = (unsigned)l2; lw[1] = dead ? 1u : 0u; } \
        asm volatile("s_waitcnt vmcnt(0) lgkmcnt(0)" ::: "memory"); \
      } \
      asm volatile("" ::: "memory"); BAR; asm volatile("" ::: "memory"); \
      lvl = __builtin_amdgcn_readfirstlane(lw[0]); bad = bad || (__builtin_amdgcn_readfirstlane(lw[1]) != 0u); \
      asm volatile("s_waitcnt lgkmcnt(0)" ::: "memory"); \
      WACC_OUT; \
    } } while (0)

  float scale = __builtin_bit_cast(float, __builtin_amdgcn_readfirstlane(__builtin_bit_cast(int, P.sx[0] * P.sw[0])));
  float lo = 0.f;
  const __amdgpu_buffer_rsrc_t orsrc_real = __builtin_amdgcn_make_buffer_rsrc(P.out, 0, (unsigned)((size_t)M * N * 4), 0x00020000);
  if (blockIdx.x >= NCOMM) {
  Unit cur, nxt; int ui = 0;
  tile_of(xg, slot, 0, cur);
  f32x4 acc[2][4][2][2];
  i32x4 At[2][2], Bt[2][2][2];
  const size_t hstep = HBSTRIDE, tstep = 2 * HBSTRIDE;
  const char* cA = As + (size_t)cur.pm * tstep; const char* cB = Bs + (size_t)cur.pn * tstep;
  __amdgpu_buffer_rsrc_t orsrc = __builtin_amdgcn_make_buffer_rsrc(P.out, 0, 0u, 0x00020000);
  WAIT_STAGE(0);
  STAMP(1);
  STAGE(SB(0, 0), cB); STAGE(SB(0, 1), cB + hstep); STAGE(SA(0, 0), cA); STAGE(SA(0, 1), cA + hstep);
  STAGE(SB(1, 0), cB + KOFF(1)); STAGE(SB(1, 1), cB + hstep + KOFF(1)); STAGE(SA(1, 0), cA + KOFF(1));
  if (wr == 1) BAR;
  WAIT_V(6); BAR; BAR;
  unsigned psb = 0;
#pragma unroll
  for (int a = 0; a < 2; ++a)
#pragma unroll
    for (int m = 0; m < 4; ++m)
#pragma unroll
      for (int h = 0; h < 2; ++h)
#pragma unroll
        for (int n = 0; n < 2; ++n) acc[a][m][h][n] = (f32x4){0.f, 0.f, 0.f, 0.f};
#define ITER(FIRST, LAST, a1, a2, b2, a3, b3) do { \
        LDB(0, 0); SCHED; LDB(0, 1); LDA(0, 0, 0); \
      if (FIRST) { ST(0, 1, 4, 8, psb); } \
      WAIT_L(8); BAR; WAIT_L(0); MMA(0, 0, FIRST); BAR; SCHED; \
        LDA(0, 0, 1); STAGE(SB(0, 0), b2); STAGE(SA(1, 1), (a1) + hstep); \
      if (FIRST) { ST(1, 0, 4, 8, psb); WAIT_V(24); } else WAIT_V(8); \
      BAR; WAIT_L(0); MMA(0, 1, FIRST); BAR; SCHED; \
        LDA(0, 1, 0); STAGE(SB(0, 1), (b2) + hstep); \
      if (FIRST) { ST(1, 1, 0, 4, psb); } \
      BAR; WAIT_L(0); MMA(1, 0, FIRST); BAR; SCHED; \
        LDA(0, 1, 1); STAGE(SA(0, 0), a2); \
      if (FIRST) { ST(1, 1, 4, 8, psb); WAIT_V(24); } else WAIT_V(8); \
      BAR; WAIT_L(0); MMA(1, 1, FIRST); BAR; SCHED; \
        LDB(1, 0); SCHED; LDB(1, 1); LDA(1, 0, 0); \
      WAIT_L(8); BAR; WAIT_L(0); MMA(0, 0, 0); BAR; SCHED; \
        LDA(1, 0, 1); STAGE(SB(1, 0), b3); STAGE(SA(0, 1), (a2) + hstep); \
      if (LAST) { ST(0, 0, 0, 4, sb); WAIT_V(12); } else if (FIRST) WAIT_V(16); else WAIT_V(8); \
      BAR; WAIT_L(0); MMA(0, 1, 0); BAR; SCHED; \
        LDA(1, 1, 0); STAGE(SB(1, 1), (b3) + hstep); \
      if (LAST) { ST(0, 0, 4, 8, sb); } \
      BAR; WAIT_L(0); MMA(1, 0, 0); BAR; SCHED; \
        LDA(1, 1, 1); STAGE(SA(1, 0), a3); \
      if (LAST) { ST(0, 1, 0, 4, sb); ST(1, 0, 0, 4, sb); WAIT_V(24); } else WAIT_V(8); \
      BAR; WAIT_L(0); MMA(1, 1, 0); BAR; SCHED; \
    } while (0)
  for (;;) {
    const bool has_next = tile_of(xg, slot, ui + 1, nxt);
    const char* nA = has_next ? As + (size_t)nxt.pm * tstep : cA; const char* nB = has_next ? Bs + (size_t)nxt.pn * tstep : cB;
    const unsigned sb = (unsigned)(cur.pm * BM * N + cur.pn * BM) * 4u;
    STAMP(30 + ui * 4);
    ITER(1, 0, cA + KOFF(1), cA + KOFF(2), cB + KOFF(2), cA + KOFF(3), cB + KOFF(3));
    STAMP(31 + ui * 4);
    orsrc = orsrc_real;
    for (int t = 2; t < NT - 2; t += 2) {
      if (((t + 2) & 7) == 0) { WAIT_STAGE(ui * 4 + ((t + 2) >> 3)); }
      if (t == 4) STAMP(32 + ui * 4);
      ITER(0, 0, cA + KOFF(t + 1), cA + KOFF(t + 2), cB + KOFF(t + 2), cA + KOFF(t + 3), cB + KOFF(t + 3));
    }
    STAMP(2 + ui * 2);
    if (has_next) WAIT_STAGE((ui + 1) * 4);
    STAMP(3 + ui * 2);
    WSTAMP(ui);
    if (bad) { scale = __builtin_nanf(""); lo = scale; }
    ITER(0, 1, cA + KOFF(NT - 1), nA, nB, nA + KOFF(1), nB + KOFF(1));
    psb = sb;
    if (!has_next) break;
    cur = nxt; cA = nA; cB = nB; ++ui;
  }
  ST(0, 1, 4, 8, psb); ST(1, 0, 4, 8, psb); ST(1, 1, 0, 8, psb);
  if (wr == 0) BAR;
  BAR;
  STAMP(20);
  }
  {
    WAIT_STAGE(NSTAGE - 1);
    if (bad) { scale = __builtin_nanf(""); lo = scale; }
    const int b = blockIdx.x, g8 = b & 7, j = b >> 3;
    const int pm = (g8 & 3) * 4 + (j & 3), pn = 60 + (g8 >> 2) * 2 + ((j >> 2) & 1), ai = (j >> 3) & 1, cb = (j >> 4) & 1;
    const int wr4 = wid >> 1, wc2 = wid & 1;
    const char* pA = As + (size_t)(pm * 2 + ai) * HBSTRIDE + voff;
    const char* pB0 = Bs + (size_t)(pn * 2) * HBSTRIDE + cb * 8192 + voff;
    const char* pB1 = pB0 + HBSTRIDE;
#define PSTAGE(t) do { const unsigned _sl = (unsigned)((t) & 3) * 32768u + ldsw; const size_t _ko = KOFF((t) & 31); \
      __builtin_amdgcn_global_load_lds((const unsigned*)(pA + _ko), (LAS unsigned*)(lds + _sl), 16, 0, 0); \
      __builtin_amdgcn_global_load_lds((const unsigned*)(pA + _ko + 8192), (LAS unsigned*)(lds + _sl + 8192), 16, 0, 0); \
      __builtin_amdgcn_global_load_lds((const unsigned*)(pB0 + _ko), (LAS unsigned*)(lds + _sl + 16384), 16, 0, 0); \
      __builtin_amdgcn_global_load_lds((const unsigned*)(pB1 + _ko), (LAS unsigned*)(lds + _sl + 24576), 16, 0, 0); } while (0)
    f32x4 pacc[2][2][2];
#pragma unroll
    for (int m = 0; m < 2; ++m)
#pragma unroll
      for (int h = 0; h < 2; ++h)
#pragma unroll
        for (int n = 0; n < 2; ++n) pacc[m][h][n] = (f32x4){0.f, 0.f, 0.f, 0.f};
    const int paoff = wr4 * 2 * 2048 + lane * 16, pboff = 16384 + wc2 * 2 * 2048 + lane * 16;
#define PLD(dstA, dstB, t) do { const unsigned _sl = (unsigned)((t) & 3) * 32768u; \
      _Pragma("unroll") for (int m = 0; m < 2; ++m) _Pragma("unroll") for (int k = 0; k < 2; ++k) dstA[m][k] = *(const LAS i32x4*)(lds + _sl + paoff + m * 2048 + k * 1024); \
      _Pragma("unroll") for (int h = 0; h < 2; ++h) _Pragma("unroll") for (int n = 0; n < 2; ++n) _Pragma("unroll") for (int k = 0; k < 2; ++k) dstB[h][n][k] = *(const LAS i32x4*)(lds + _sl + pboff + h * 8192 + n * 2048 + k * 1024); } while (0)
#define PMMA(srcA, srcB) do { _Pragma("unroll") for (int m = 0; m < 2; ++m) _Pragma("unroll") for (int h = 0; h < 2; ++h) _Pragma("unroll") for (int n = 0; n < 2; ++n) \
      asm volatile("v_mfma_f32_16x16x128_f8f6f4 %0, %1, %2, %0" : "+v"(pacc[m][h][n]) : "v"(CAT(srcA[m])), "v"(CAT(srcB[h][n]))); } while (0)
    i32x4 pa0[2][2], pb0[2][2][2], pa1[2][2], pb1[2][2][2];
    PSTAGE(0); PSTAGE(1); PSTAGE(2);
    WAIT_V(8); BAR; PLD(pa0, pb0, 0);
    for (int t = 0; t < NT; t += 2) {
      WAIT_V(4); WAIT_L(0); BAR;
      PSTAGE(t + 3); PLD(pa1, pb1, t + 1); SCHED;
      PMMA(pa0, pb0);
      WAIT_V(4); WAIT_L(0); BAR;
      PSTAGE(t + 4); PLD(pa0, pb0, t + 2); SCHED;
      PMMA(pa1, pb1);
    }
    asm volatile("s_nop 15\n\ts_nop 15" ::: "memory");
    const unsigned pov = (unsigned)((pm * 256 + ai * 128 + wr4 * 32 + fq * 4) * N + pn * 256 + (cb * 2 + wc2) * 64 + fr * 4) * 4u;
#pragma unroll
    for (int m = 0; m < 2; ++m)
#pragma unroll
      for (int jj = 0; jj < 4; ++jj) {
        f32x4 v = {pacc[m][0][0][jj], pacc[m][0][1][jj], pacc[m][1][0][jj], pacc[m][1][1][jj]}; v = v * scale; v = __builtin_elementwise_max(v, (f32x4){lo, lo, lo, lo});
        __builtin_amdgcn_raw_buffer_store_b128(__builtin_bit_cast(u32x4, v), orsrc_real, pov + (unsigned)((m * 16 + jj) * N) * 4u, 0, 2); }
    WAIT_V(0);
  }
  STAMP(63);
}
}

extern "C" void kernel_launch(int rank, int nranks, ncclComm_t comm,
                              hipStream_t stream,
                              void* const* d_in, const int* in_sizes, int n_in,
                              void* d_out, int out_size,
                              const int* in_dtypes, int out_dtype,
                              void* const* d_out_all,
                              void* const* d_ws_all, size_t ws_bytes) {
  if (nranks != NR || n_in < 4 || ws_bytes < CTL_OFF + CTL_BYTES || (unsigned)rank >= (unsigned)NR) { fprintf(stderr, "unsupported configuration\n"); return; }
  static bool init = false;
  if (!init) {
    HIP_CHECK(hipFuncSetAttribute((const void*)fused, hipFuncAttributeMaxDynamicSharedMemorySize, LDS_BYTES));
    init = true;
  }
  Params P;
  P.x = (const unsigned char*)d_in[0]; P.w = (const unsigned char*)d_in[1];
  P.sx = (const float*)d_in[2]; P.sw = (const float*)d_in[3];
  P.out = (float*)d_out; P.rank = rank;
  for (int p = 0; p < NR; ++p) P.ws[p] = (char*)d_ws_all[p];
  hipLaunchKernelGGL(fused, dim3(GRID), dim3(512), LDS_BYTES, stream, P);
}
```

```cpp
#include <hip/hip_runtime.h>
#include <rccl/rccl.h>
#include <cstdio>

#define HIP_CHECK(expr) do { hipError_t _e = (expr); if (_e != hipSuccess) { \
    fprintf(stderr, "HIP error %s at %s:%d\n", hipGetErrorString(_e), __FILE__, __LINE__); \
    return; } } while (0)

namespace {
#define LAS __attribute__((address_space(3)))
typedef float f32x4 __attribute__((ext_vector_type(4)));
typedef int i32x4 __attribute__((ext_vector_type(4)));
typedef unsigned u32x4 __attribute__((ext_vector_type(4)));
typedef unsigned u32x2 __attribute__((ext_vector_type(2)));

constexpr int NR = 8;
constexpr int M = 4096, N = 16384, K = 4096, KS = K / NR;
constexpr int BM = 256, HTB = 16384, STAGE_BYTES = 8 * HTB, NT = K / 128;
constexpr int LDS_BYTES = STAGE_BYTES + 256;
constexpr size_t HBSTRIDE = (size_t)4 * HTB;
constexpr size_t XS_BYTES = (size_t)M * KS;
constexpr size_t SRC_STRIDE = (size_t)(M + N) * KS;
constexpr size_t BUF_BYTES = NR * SRC_STRIDE;
constexpr size_t CTL_OFF = 2 * BUF_BYTES;
#define NCOMM_ 16
constexpr int NCOMM = NCOMM_;
constexpr int GRID = 256;
constexpr int WPX = (GRID - NCOMM) / 8;
constexpr int NROUND = 5;
constexpr int NSTAGE = 17;
constexpr int NUNIT = 320;
constexpr size_t CTL_FLAGS = 0;
constexpr size_t CTL_CNT = 64 * 1024;
constexpr size_t CTL_TMO = 68 * 1024;
constexpr size_t CTL_BYTES = 72 * 1024;
#define STAMP(i) do { } while (0)
#define STAMPW(i) do { } while (0)
#define WACC_IN do { } while (0)
#define WACC_OUT do { } while (0)
#define WSTAMP(i) do { } while (0)
#define COMM_HF 1
#define COMM_B 2
#define DRAIN_END 140
#define COMM_AUX 17
#define STAUX0 2
#define STAUX1 0
#define COMM_Q 6
#define COMM_ROT 0
#define COMM_NDEST 8
#define POLL_SLEEP 8
constexpr unsigned long long DEADLINE_TICKS = 40000000ull;

struct Params {
  const unsigned char* x;
  const unsigned char* w;
  const float* sx; const float* sw;
  float* out;
  char* ws[NR];
  int rank;
};

__host__ __device__ constexpr int round_col0(int r) { return 15 * r; }
__host__ __device__ constexpr int round_ncol(int r) { return r < 4 ? 15 : 4; }

__device__ __forceinline__ int level_of(int B) { return B < 124 ? B / 31 : (B < 140 ? 4 : (B >= NUNIT ? NSTAGE : 5 + (B - 140) / 15)); }
__device__ __forceinline__ int stage_of(int r, int kq) { return r == 0 ? kq : 4 * r + 1 + kq; }
struct Unit { int pm, pn; };
__device__ __forceinline__ bool tile_of(int g, int j, int r, Unit& u) {
  const int band = g & 3, h = g >> 2;
  if (r < 4) { const int i = 30 * (2 * r + h) + j; u.pm = band * 4 + (i & 3); u.pn = i >> 2; return true; }
  return false;
}

__device__ __forceinline__ unsigned perm(unsigned a, unsigned b, unsigned sel) { return __builtin_amdgcn_perm(a, b, sel); }

__global__ __launch_bounds__(512, 2) void fused(Params P) {
  extern __shared__ __attribute__((aligned(16))) unsigned char shm[];
  LAS unsigned char* lds = (LAS unsigned char*)shm;
  const int tid = threadIdx.x, wid = __builtin_amdgcn_readfirstlane(tid >> 6), lane = tid & 63;
  char* const myws = P.ws[P.rank];
  STAMP(0);
  char* const ctl = myws + CTL_OFF;
  unsigned call;
  {
    unsigned* cnt = (unsigned*)(ctl + CTL_CNT) + blockIdx.x;
    LAS unsigned* l = (LAS unsigned*)(lds + STAGE_BYTES);
    if (tid == 0) { unsigned c = __hip_atomic_load(cnt, __ATOMIC_RELAXED, __HIP_MEMORY_SCOPE_AGENT) + 1u; __hip_atomic_store(cnt, c, __ATOMIC_RELAXED, __HIP_MEMORY_SCOPE_AGENT); l[0] = c; }
    __syncthreads();
    call = __builtin_amdgcn_readfirstlane(l[0]);
    __syncthreads();
  }
  const size_t bufoff = (call & 1u) ? BUF_BYTES : 0;
  unsigned* const tmo = (unsigned*)(ctl + CTL_TMO);

  if (blockIdx.x < NCOMM) {
    const int cj = blockIdx.x;
    const int rot = COMM_ROT;
    const int fr = lane & 15, fq = lane >> 4;
    u32x4 d[4];
    struct UD { int kq, isx, pan; };
    auto decode = [](int g) -> UD {
      UD u;
      const int a = g / 31, ai = g - a * 31;
      const int b = (g - 140) / 15, bi = (g - 140) - b * 15;
      const int c = g - 124;
      const bool r0 = g < 124, lf = !r0 && g < 140;
      u.kq = r0 ? a : (lf ? (c >> 2) : (b & 3));
      u.isx = r0 && ai < 16;
      u.pan = r0 ? (ai < 16 ? ai : ai - 16) : (lf ? 60 + (c & 3) : 15 * (1 + (b >> 2)) + bi);
      return u; };
    auto load_unit = [&](int g) {
      const UD u = decode(g); const int kq = u.kq;
      if (u.isx) {
#pragma unroll
        for (int i = 0; i < 4; ++i) { const int p = tid + 512 * i, hb = p >> 10, gg = (p >> 7) & 7, h = (p >> 6) & 1;
          const int row = u.pan * 256 + hb * 128 + gg * 16 + fr;
          d[i] = *(const u32x4*)(P.x + (size_t)row * KS + kq * 128 + h * 64 + fq * 16); }
      } else {
        const int wc = wid >> 1, h = wid & 1;
        const unsigned char* src = P.w + (size_t)(kq * 128 + h * 64 + fq * 16) * N + u.pan * 256 + wc * 64 + fr * 4;
#pragma unroll
        for (int j = 0; j < 16; ++j) d[j >> 2][j & 3] = *(const unsigned*)(src + (size_t)j * N);
      } };
    auto publish = [&](int n) {
      asm volatile("" ::: "memory"); __builtin_amdgcn_s_barrier(); asm volatile("" ::: "memory");
      if (wid == 0 && lane < NR) {
        unsigned* f = (unsigned*)(P.ws[lane] + CTL_OFF + CTL_FLAGS + (size_t)P.rank * 128) + cj;
        __hip_atomic_store(f, call * 32u + (unsigned)n, __ATOMIC_RELAXED, __HIP_MEMORY_SCOPE_SYSTEM); } };
    bool drained = false;
    int g = cj; load_unit(g);
    for (int it = 0; g < NUNIT; ++it, g += NCOMM) {
      const UD u = decode(g);
      const unsigned sbase_ = (unsigned)(P.rank * SRC_STRIDE) + (unsigned)(u.kq * HTB);
      u32x4 v[4]; unsigned off[4];
      if (u.isx) {
#pragma unroll
        for (int i = 0; i < 4; ++i) { const int p = tid + 512 * i, hb = p >> 10; v[i] = d[i];
          off[i] = sbase_ + (unsigned)((u.pan * 2 + hb) * HBSTRIDE) + (unsigned)((p & 1023) * 16); }
      } else {
        const int wc = wid >> 1, h = wid & 1;
#pragma unroll
        for (int q4 = 0; q4 < 4; ++q4) {
          const unsigned a0 = d[q4][0], a1 = d[q4][1], a2 = d[q4][2], a3 = d[q4][3];
          const unsigned t0 = perm(a1, a0, 0x05010400u), t1 = perm(a1, a0, 0x07030602u);
          const unsigned t2 = perm(a3, a2, 0x05010400u), t3 = perm(a3, a2, 0x07030602u);
          v[0][q4] = perm(t2, t0, 0x05040100u); v[1][q4] = perm(t2, t0, 0x07060302u);
          v[2][q4] = perm(t3, t1, 0x05040100u); v[3][q4] = perm(t3, t1, 0x07060302u);
        }
#pragma unroll
        for (int c = 0; c < 4; ++c) { const int hb = c >> 1, n = c & 1;
          off[c] = sbase_ + (unsigned)XS_BYTES + (unsigned)((u.pan * 2 + hb) * HBSTRIDE) + (unsigned)((wc * 2 + n) * 2048 + h * 1024 + lane * 16); }
      }
      if (g + NCOMM < NUNIT) load_unit(g + NCOMM);
#pragma unroll
      for (int hf = 0; hf < 2; ++hf) {
#pragma unroll
        for (int q = 0; q < NR; ++q) {
          const int pd = (P.rank + 1 + q + rot) & (NR - 1);
          const __amdgpu_buffer_rsrc_t rsq = __builtin_amdgcn_make_buffer_rsrc(P.ws[pd] + bufoff, 0, (unsigned)BUF_BYTES, 0x00020000);
#pragma unroll
          for (int i = 0; i < 2; ++i) __builtin_amdgcn_raw_buffer_store_b128(v[hf * 2 + i], rsq, off[hf * 2 + i], 0, COMM_AUX);
          if (hf == COMM_HF && q == 6 && it > 0 && !drained) { asm volatile("s_waitcnt vmcnt(%0)" :: "n"(16 * COMM_HF + 14) : "memory"); publish(it); STAMP(it); }
        }
      }
      drained = false;
      if (g >= 124 - NCOMM && g < DRAIN_END) { asm volatile("s_waitcnt vmcnt(0)" ::: "memory"); publish(it + 1); drained = true; }
    }
    asm volatile("s_waitcnt vmcnt(0)" ::: "memory");
    publish(NUNIT / NCOMM);
    STAMP(40);
  }

  const int wr = wid >> 2, wc = wid & 3, fr = lane & 15, fq = lane >> 4;
  const int gi = blockIdx.x - NCOMM, xg = gi & 7, slot = gi >> 3;
  const char* const As = myws + bufoff; const char* const Bs = As + XS_BYTES;
  const unsigned voff = tid * 16;
  const unsigned ldsw = (unsigned)wid * 1024u;
  const int aoff = wr * 4 * 2048 + lane * 16, boff = wc * 2 * 2048 + lane * 16;
  const unsigned ovoff = (unsigned)((wr * 64 + fq * 4) * N + wc * 64 + fr * 4) * 4u;
  int lvl = 0;
  bool bad = false;
#define SA(b, h) (((b) * 2 + (h)) * HTB)
#define SB(b, h) ((4 + (b) * 2 + (h)) * HTB)
#define STAGE(bufoff_, gbase) do { _Pragma("unroll") for (int _i = 0; _i < 2; ++_i) \
        __builtin_amdgcn_global_load_lds((const unsigned*)((const char*)(gbase) + voff + _i * 8192), (LAS unsigned*)(lds + (bufoff_) + ldsw + _i * 8192), 16, 0, 0); } while (0)
#define LDA(b, h, mp) do { _Pragma("unroll") for (int m = 0; m < 2; ++m) _Pragma("unroll") for (int k = 0; k < 2; ++k) At[m][k] = *(const LAS i32x4*)(lds + SA(b, h) + aoff + ((mp) * 2 + m) * 2048 + k * 1024); } while (0)
#define LDB(b, h) do { _Pragma("unroll") for (int n = 0; n < 2; ++n) _Pragma("unroll") for (int k = 0; k < 2; ++k) Bt[h][n][k] = *(const LAS i32x4*)(lds + SB(b, h) + boff + n * 2048 + k * 1024); } while (0)
#define CAT(x) __builtin_shufflevector(x[0], x[1], 0, 1, 2, 3, 4, 5, 6, 7)
#define MMA(ai, mp, ZERO) do { __builtin_amdgcn_s_setprio(1); _Pragma("unroll") for (int m = 0; m < 2; ++m) _Pragma("unroll") for (int h = 0; h < 2; ++h) _Pragma("unroll") for (int n = 0; n < 2; ++n) { \
        if (ZERO) asm volatile("v_mfma_f32_16x16x128_f8f6f4 %0, %1, %2, 0" : "=v"(acc[ai][(mp) * 2 + m][h][n]) : "v"(CAT(At[m])), "v"(CAT(Bt[h][n]))); \
        else asm volatile("v_mfma_f32_16x16x128_f8f6f4 %0, %1, %2, %0" : "+v"(acc[ai][(mp) * 2 + m][h][n]) : "v"(CAT(At[m])), "v"(CAT(Bt[h][n]))); } \
        __builtin_amdgcn_s_setprio(0); } while (0)
#define ST(ai, mp, s0, s1, sbase) do { _Pragma("unroll") for (int s = (s0); s < (s1); ++s) { const int m = (mp) * 2 + (s >> 2), j = s & 3; \
        f32x4 v = {acc[ai][m][0][0][j], acc[ai][m][0][1][j], acc[ai][m][1][0][j], acc[ai][m][1][1][j]}; v = v * scale; v = __builtin_elementwise_max(v, (f32x4){lo, lo, lo, lo}); \
        __builtin_amdgcn_raw_buffer_store_b128(__builtin_bit_cast(u32x4, v), orsrc, ovoff + ((sbase) + (unsigned)(((ai) * 128 + m * 16 + j) * N) * 4u), 0, ((ai) == 0) ? STAUX0 : STAUX1); } } while (0)
#define WAIT_V(n) asm volatile("s_waitcnt vmcnt(" #n ")" ::: "memory")
#define WAIT_L(n) asm volatile("s_waitcnt lgkmcnt(" #n ")" ::: "memory")
#define BAR __builtin_amdgcn_s_barrier()
#define SCHED __builtin_amdgcn_sched_barrier(0)
#define KOFF(t) ((size_t)((t) & 7) * SRC_STRIDE + (size_t)((t) >> 3) * HTB)
  auto level_from = [&](unsigned long long f0) -> int {
    const unsigned base = call * 32u; int B = 1 << 20;
#pragma unroll
    for (int k = 0; k < 2; ++k) {
      const unsigned v0 = (unsigned)(f0 >> (32 * k));
      const int n = (v0 > base && v0 - base < 32u) ? (int)(v0 - base) : 0;
      const int b = (n >= NUNIT / NCOMM) ? (1 << 20) : (lane & 7) * 2 + k + NCOMM * n;
      B = min(B, b);
    }
#pragma unroll
    for (int o = 32; o >= 1; o >>= 1) B = min(B, __shfl_xor(B, o));
    return level_of(__builtin_amdgcn_readfirstlane(B)); };
  const char* const fb = ctl + CTL_FLAGS + (lane >> 3) * 128 + (lane & 7) * 8;
  auto probe_level = [&]() -> int { return level_from(__hip_atomic_load((const unsigned long long*)fb, __ATOMIC_RELAXED, __HIP_MEMORY_SCOPE_SYSTEM)); };
#define WAIT_STAGE(s) do { if ((s) >= lvl) { \
      LAS unsigned* lw = (LAS unsigned*)(lds + STAGE_BYTES); \
      WACC_IN; \
      if (wid == 0) { \
        int l2; const unsigned long long t0 = __builtin_amdgcn_s_memrealtime(); bool dead = false; \
        for (;;) { \
          l2 = probe_level(); \
          if (l2 > (s)) break; \
          if (__hip_atomic_load(tmo, __ATOMIC_RELAXED, __HIP_MEMORY_SCOPE_AGENT) == call || __builtin_amdgcn_s_memrealtime() - t0 > DEADLINE_TICKS) { \
            if (lane == 0) __hip_atomic_store(tmo, call, __ATOMIC_RELAXED, __HIP_MEMORY_SCOPE_AGENT); \
            dead = true; l2 = NSTAGE; break; } \
          __builtin_amdgcn_s_sleep(POLL_SLEEP); \
        } \
        __builtin_amdgcn_fence(__ATOMIC_ACQUIRE, ""); \
        if (lane == 0) { lw[0] = (unsigned)l2; lw[1] = dead ? 1u : 0u; } \
        asm volatile("s_waitcnt vmcnt(0) lgkmcnt(0)" ::: "memory"); \
      } \
      asm volatile("" ::: "memory"); BAR; asm volatile("" ::: "memory"); \
      lvl = __builtin_amdgcn_readfirstlane(lw[0]); bad = bad || (__builtin_amdgcn_readfirstlane(lw[1]) != 0u); \
      asm volatile("s_waitcnt lgkmcnt(0)" ::: "memory"); \
      WACC_OUT; \
    } } while (0)

  float scale = __builtin_bit_cast(float, __builtin_amdgcn_readfirstlane(__builtin_bit_cast(int, P.sx[0] * P.sw[0])));
  float lo = 0.f;
  const __amdgpu_buffer_rsrc_t orsrc_real = __builtin_amdgcn_make_buffer_rsrc(P.out, 0, (unsigned)((size_t)M * N * 4), 0x00020000);
  for (int sess = 0; sess < 2; ++sess) {
  const int r_begin = sess, r_end = sess == 0 ? 1 : 4;
  if (blockIdx.x >= NCOMM) {
  Unit cur, nxt; int ui = r_begin;
  tile_of(xg, slot, ui, cur);
  f32x4 acc[2][4][2][2];
  i32x4 At[2][2], Bt[2][2][2];
  const size_t hstep = HBSTRIDE, tstep = 2 * HBSTRIDE;
  const char* cA = As + (size_t)cur.pm * tstep; const char* cB = Bs + (size_t)cur.pn * tstep;
  __amdgpu_buffer_rsrc_t orsrc = __builtin_amdgcn_make_buffer_rsrc(P.out, 0, 0u, 0x00020000);
  WAIT_STAGE(stage_of(r_begin, 0));
  STAMP(1 + 20 * sess);
  STAGE(SB(0, 0), cB); STAGE(SB(0, 1), cB + hstep); STAGE(SA(0, 0), cA); STAGE(SA(0, 1), cA + hstep);
  STAGE(SB(1, 0), cB + KOFF(1)); STAGE(SB(1, 1), cB + hstep + KOFF(1)); STAGE(SA(1, 0), cA + KOFF(1));
  if (wr == 1) BAR;
  WAIT_V(6); BAR; BAR;
  unsigned psb = 0;
#pragma unroll
  for (int a = 0; a < 2; ++a)
#pragma unroll
    for (int m = 0; m < 4; ++m)
#pragma unroll
      for (int h = 0; h < 2; ++h)
#pragma unroll
        for (int n = 0; n < 2; ++n) acc[a][m][h][n] = (f32x4){0.f, 0.f, 0.f, 0.f};
#define ITER(FIRST, LAST, a1, a2, b2, a3, b3) do { \
        LDB(0, 0); SCHED; LDB(0, 1); LDA(0, 0, 0); \
      if (FIRST) { ST(0, 1, 4, 8, psb); } \
      WAIT_L(8); BAR; WAIT_L(0); MMA(0, 0, FIRST); BAR; SCHED; \
        LDA(0, 0, 1); STAGE(SB(0, 0), b2); STAGE(SA(1, 1), (a1) + hstep); \
      if (FIRST) { ST(1, 0, 4, 8, psb); WAIT_V(24); } else WAIT_V(8); \
      BAR; WAIT_L(0); MMA(0, 1, FIRST); BAR; SCHED; \
        LDA(0, 1, 0); STAGE(SB(0, 1), (b2) + hstep); \
      if (FIRST) { ST(1, 1, 0, 4, psb); } \
      BAR; WAIT_L(0); MMA(1, 0, FIRST); BAR; SCHED; \
        LDA(0, 1, 1); STAGE(SA(0, 0), a2); \
      if (FIRST) { ST(1, 1, 4, 8, psb); WAIT_V(24); } else WAIT_V(8); \
      BAR; WAIT_L(0); MMA(1, 1, FIRST); BAR; SCHED; \
        LDB(1, 0); SCHED; LDB(1, 1); LDA(1, 0, 0); \
      WAIT_L(8); BAR; WAIT_L(0); MMA(0, 0, 0); BAR; SCHED; \
        LDA(1, 0, 1); STAGE(SB(1, 0), b3); STAGE(SA(0, 1), (a2) + hstep); \
      if (LAST) { ST(0, 0, 0, 4, sb); WAIT_V(12); } else if (FIRST) WAIT_V(16); else WAIT_V(8); \
      BAR; WAIT_L(0); MMA(0, 1, 0); BAR; SCHED; \
        LDA(1, 1, 0); STAGE(SB(1, 1), (b3) + hstep); \
      if (LAST) { ST(0, 0, 4, 8, sb); } \
      BAR; WAIT_L(0); MMA(1, 0, 0); BAR; SCHED; \
        LDA(1, 1, 1); STAGE(SA(1, 0), a3); \
      if (LAST) { ST(0, 1, 0, 4, sb); ST(1, 0, 0, 4, sb); WAIT_V(24); } else WAIT_V(8); \
      BAR; WAIT_L(0); MMA(1, 1, 0); BAR; SCHED; \
    } while (0)
  for (;;) {
    const bool has_next = (ui + 1 < r_end) && tile_of(xg, slot, ui + 1, nxt);
    const char* nA = has_next ? As + (size_t)nxt.pm * tstep : cA; const char* nB = has_next ? Bs + (size_t)nxt.pn * tstep : cB;
    const unsigned sb = (unsigned)(cur.pm * BM * N + cur.pn * BM) * 4u;
    ITER(1, 0, cA + KOFF(1), cA + KOFF(2), cB + KOFF(2), cA + KOFF(3), cB + KOFF(3));
    orsrc = orsrc_real;
    for (int t = 2; t < NT - 2; t += 2) {
      if (((t + 2) & 7) == 0) { WAIT_STAGE(stage_of(ui, (t + 2) >> 3)); }
      ITER(0, 0, cA + KOFF(t + 1), cA + KOFF(t + 2), cB + KOFF(t + 2), cA + KOFF(t + 3), cB + KOFF(t + 3));
    }
    STAMP(2 + ui * 2);
    if (has_next) WAIT_STAGE(stage_of(ui + 1, 0));
    if (bad) { scale = __builtin_nanf(""); lo = scale; }
    ITER(0, 1, cA + KOFF(NT - 1), nA, nB, nA + KOFF(1), nB + KOFF(1));
    psb = sb;
    if (!has_next) break;
    cur = nxt; cA = nA; cB = nB; ++ui;
  }
  ST(0, 1, 4, 8, psb); ST(1, 0, 4, 8, psb); ST(1, 1, 0, 8, psb);
  if (wr == 0) BAR;
  BAR;
  STAMP(20 + sess);
  }
  if (sess == 0) {
    WAIT_STAGE(4);
    if (bad) { scale = __builtin_nanf(""); lo = scale; }
    const int b = blockIdx.x, g8 = b & 7, j = b >> 3;
    const int pm = (g8 & 3) * 4 + (j & 3), pn = 60 + (g8 >> 2) * 2 + ((j >> 2) & 1), ai = (j >> 3) & 1, cb = (j >> 4) & 1;
    const int wr4 = wid >> 1, wc2 = wid & 1;
    const char* pA = As + (size_t)(pm * 2 + ai) * HBSTRIDE + voff;
    const char* pB0 = Bs + (size_t)(pn * 2) * HBSTRIDE + cb * 8192 + voff;
    const char* pB1 = pB0 + HBSTRIDE;
#define PSTAGE(t) do { const unsigned _sl = (unsigned)((t) & 3) * 32768u + ldsw; const size_t _ko = KOFF((t) & 31); \
      __builtin_amdgcn_global_load_lds((const unsigned*)(pA + _ko), (LAS unsigned*)(lds + _sl), 16, 0, 0); \
      __builtin_amdgcn_global_load_lds((const unsigned*)(pA + _ko + 8192), (LAS unsigned*)(lds + _sl + 8192), 16, 0, 0); \
      __builtin_amdgcn_global_load_lds((const unsigned*)(pB0 + _ko), (LAS unsigned*)(lds + _sl + 16384), 16, 0, 0); \
      __builtin_amdgcn_global_load_lds((const unsigned*)(pB1 + _ko), (LAS unsigned*)(lds + _sl + 24576), 16, 0, 0); } while (0)
    f32x4 pacc[2][2][2];
#pragma unroll
    for (int m = 0; m < 2; ++m)
#pragma unroll
      for (int h = 0; h < 2; ++h)
#pragma unroll
        for (int n = 0; n < 2; ++n) pacc[m][h][n] = (f32x4){0.f, 0.f, 0.f, 0.f};
    const int paoff = wr4 * 2 * 2048 + lane * 16, pboff = 16384 + wc2 * 2 * 2048 + lane * 16;
#define PLD(dstA, dstB, t) do { const unsigned _sl = (unsigned)((t) & 3) * 32768u; \
      _Pragma("unroll") for (int m = 0; m < 2; ++m) _Pragma("unroll") for (int k = 0; k < 2; ++k) dstA[m][k] = *(const LAS i32x4*)(lds + _sl + paoff + m * 2048 + k * 1024); \
      _Pragma("unroll") for (int h = 0; h < 2; ++h) _Pragma("unroll") for (int n = 0; n < 2; ++n) _Pragma("unroll") for (int k = 0; k < 2; ++k) dstB[h][n][k] = *(const LAS i32x4*)(lds + _sl + pboff + h * 8192 + n * 2048 + k * 1024); } while (0)
#define PMMA(srcA, srcB) do { _Pragma("unroll") for (int m = 0; m < 2; ++m) _Pragma("unroll") for (int h = 0; h < 2; ++h) _Pragma("unroll") for (int n = 0; n < 2; ++n) \
      asm volatile("v_mfma_f32_16x16x128_f8f6f4 %0, %1, %2, %0" : "+v"(pacc[m][h][n]) : "v"(CAT(srcA[m])), "v"(CAT(srcB[h][n]))); } while (0)
    i32x4 pa0[2][2], pb0[2][2][2], pa1[2][2], pb1[2][2][2];
    PSTAGE(0); PSTAGE(1); PSTAGE(2);
    WAIT_V(8); BAR; PLD(pa0, pb0, 0);
    for (int t = 0; t < NT; t += 2) {
      WAIT_V(4); WAIT_L(0); BAR;
      PSTAGE(t + 3); PLD(pa1, pb1, t + 1); SCHED;
      PMMA(pa0, pb0);
      WAIT_V(4); WAIT_L(0); BAR;
      PSTAGE(t + 4); PLD(pa0, pb0, t + 2); SCHED;
      PMMA(pa1, pb1);
    }
    asm volatile("s_nop 15\n\ts_nop 15" ::: "memory");
    const unsigned pov = (unsigned)((pm * 256 + ai * 128 + wr4 * 32 + fq * 4) * N + pn * 256 + (cb * 2 + wc2) * 64 + fr * 4) * 4u;
#pragma unroll
    for (int m = 0; m < 2; ++m)
#pragma unroll
      for (int jj = 0; jj < 4; ++jj) {
        f32x4 v = {pacc[m][0][0][jj], pacc[m][0][1][jj], pacc[m][1][0][jj], pacc[m][1][1][jj]}; v = v * scale; v = __builtin_elementwise_max(v, (f32x4){lo, lo, lo, lo});
        __builtin_amdgcn_raw_buffer_store_b128(__builtin_bit_cast(u32x4, v), orsrc_real, pov + (unsigned)((m * 16 + jj) * N) * 4u, 0, 2); }
    WAIT_V(0); BAR;
    STAMP(22);
  }
  }
  STAMP(63);
}
}

extern "C" void kernel_launch(int rank, int nranks, ncclComm_t comm,
                              hipStream_t stream,
                              void* const* d_in, const int* in_sizes, int n_in,
                              void* d_out, int out_size,
                              const int* in_dtypes, int out_dtype,
                              void* const* d_out_all,
                              void* const* d_ws_all, size_t ws_bytes) {
  if (nranks != NR || n_in < 4 || ws_bytes < CTL_OFF + CTL_BYTES || (unsigned)rank >= (unsigned)NR) { fprintf(stderr, "unsupported configuration\n"); return; }
  static bool init = false;
  if (!init) {
    HIP_CHECK(hipFuncSetAttribute((const void*)fused, hipFuncAttributeMaxDynamicSharedMemorySize, LDS_BYTES));
    init = true;
  }
  Params P;
  P.x = (const unsigned char*)d_in[0]; P.w = (const unsigned char*)d_in[1];
  P.sx = (const float*)d_in[2]; P.sw = (const float*)d_in[3];
  P.out = (float*)d_out; P.rank = rank;
  for (int p = 0; p < NR; ++p) P.ws[p] = (char*)d_ws_all[p];
  hipLaunchKernelGGL(fused, dim3(GRID), dim3(512), LDS_BYTES, stream, P);
}
```

```cpp
#include <hip/hip_runtime.h>
#include <rccl/rccl.h>
#include <cstdio>

#define HIP_CHECK(expr) do { hipError_t _e = (expr); if (_e != hipSuccess) { \
    fprintf(stderr, "HIP error %s at %s:%d\n", hipGetErrorString(_e), __FILE__, __LINE__); \
    return; } } while (0)

namespace {
#define LAS __attribute__((address_space(3)))
typedef float f32x4 __attribute__((ext_vector_type(4)));
typedef int i32x4 __attribute__((ext_vector_type(4)));
typedef unsigned u32x4 __attribute__((ext_vector_type(4)));
typedef unsigned u32x2 __attribute__((ext_vector_type(2)));

constexpr int NR = 8;
constexpr int M = 4096, N = 16384, K = 4096, KS = K / NR;
constexpr int BM = 256, HTB = 16384, STAGE_BYTES = 8 * HTB, NT = K / 128;
constexpr int LDS_BYTES = STAGE_BYTES + 256;
constexpr size_t HBSTRIDE = (size_t)4 * HTB;
constexpr size_t XS_BYTES = (size_t)M * KS;
constexpr size_t SRC_STRIDE = (size_t)(M + N) * KS;
constexpr size_t BUF_BYTES = NR * SRC_STRIDE;
constexpr size_t CTL_OFF = 2 * BUF_BYTES;
#define NCOMM_ 16
constexpr int NCOMM = NCOMM_;
constexpr int GRID = 256;
constexpr int WPX = (GRID - NCOMM) / 8;
constexpr int NROUND = 5;
constexpr int NSTAGE = 20;
constexpr int NUNIT = 320;
constexpr size_t CTL_FLAGS = 0;
constexpr size_t CTL_CNT = 64 * 1024;
constexpr size_t CTL_TMO = 68 * 1024;
constexpr size_t CTL_BYTES = 72 * 1024;
#define STAMP(i) do { } while (0)
#define STAMPW(i) do { } while (0)
#define WACC_IN do { } while (0)
#define WACC_OUT do { } while (0)
#define WSTAMP(i) do { } while (0)
#define COMM_HF 1
#define COMM_B 2
#define DRAIN_END 140
#define DRAIN_BEG 124
#define COMM_AUX 17
#define STAUX0 2
#define STAUX1 0
#define COMM_Q 6
#define COMM_ROT 0
#define COMM_NDEST 8
#define POLL_SLEEP 8
constexpr unsigned long long DEADLINE_TICKS = 40000000ull;

struct Params {
  const unsigned char* x;
  const unsigned char* w;
  const float* sx; const float* sw;
  float* out;
  char* ws[NR];
  int rank;
};

__host__ __device__ constexpr int round_col0(int r) { return 15 * r; }
__host__ __device__ constexpr int round_ncol(int r) { return r < 4 ? 15 : 4; }

__device__ __forceinline__ int level_of(int B) { return B < 80 ? B / 20 : (B >= NUNIT ? NSTAGE : 4 + (B - 80) / 15); }
__device__ __forceinline__ int stage_of(int r, int kq) { return 4 + 4 * r + kq; }
struct Unit { int pm, pn; };
__device__ __forceinline__ bool tile_of(int g, int j, int r, Unit& u) {
  const int band = g & 3, h = g >> 2;
  if (r < 4) { const int i = 30 * (2 * r + h) + j; u.pm = band * 4 + (i & 3); u.pn = i >> 2; return true; }
  return false;
}

__device__ __forceinline__ unsigned perm(unsigned a, unsigned b, unsigned sel) { return __builtin_amdgcn_perm(a, b, sel); }

__global__ __launch_bounds__(512, 2) void fused(Params P) {
  extern __shared__ __attribute__((aligned(16))) unsigned char shm[];
  LAS unsigned char* lds = (LAS unsigned char*)shm;
  const int tid = threadIdx.x, wid = __builtin_amdgcn_readfirstlane(tid >> 6), lane = tid & 63;
  char* const myws = P.ws[P.rank];
  STAMP(0);
  char* const ctl = myws + CTL_OFF;
  unsigned call;
  {
    unsigned* cnt = (unsigned*)(ctl + CTL_CNT) + blockIdx.x;
    LAS unsigned* l = (LAS unsigned*)(lds + STAGE_BYTES);
    if (tid == 0) { unsigned c = __hip_atomic_load(cnt, __ATOMIC_RELAXED, __HIP_MEMORY_SCOPE_AGENT) + 1u; __hip_atomic_store(cnt, c, __ATOMIC_RELAXED, __HIP_MEMORY_SCOPE_AGENT); l[0] = c; }
    __syncthreads();
    call = __builtin_amdgcn_readfirstlane(l[0]);
    __syncthreads();
  }
  const size_t bufoff = (call & 1u) ? BUF_BYTES : 0;
  unsigned* const tmo = (unsigned*)(ctl + CTL_TMO);

  if (blockIdx.x < NCOMM) {
    const int cj = blockIdx.x;
    const int rot = COMM_ROT;
    const int fr = lane & 15, fq = lane >> 4;
    u32x4 d[4];
    struct UD { int kq, isx, pan; };
    auto decode = [](int g) -> UD {
      UD u;
      const int a = g / 20, ai = g - a * 20;
      const int b = (g - 80) / 15, bi = (g - 80) - b * 15;
      const bool r0 = g < 80;
      u.kq = r0 ? a : (b & 3);
      u.isx = r0 && ai < 16;
      u.pan = r0 ? (ai < 16 ? ai : 44 + ai) : 15 * (b >> 2) + bi;
      return u; };
    auto load_unit = [&](int g) {
      const UD u = decode(g); const int kq = u.kq;
      if (u.isx) {
#pragma unroll
        for (int i = 0; i < 4; ++i) { const int p = tid + 512 * i, hb = p >> 10, gg = (p >> 7) & 7, h = (p >> 6) & 1;
          const int row = u.pan * 256 + hb * 128 + gg * 16 + fr;
          d[i] = *(const u32x4*)(P.x + (size_t)row * KS + kq * 128 + h * 64 + fq * 16); }
      } else {
        const int wc = wid >> 1, h = wid & 1;
        const unsigned char* src = P.w + (size_t)(kq * 128 + h * 64 + fq * 16) * N + u.pan * 256 + wc * 64 + fr * 4;
#pragma unroll
        for (int j = 0; j < 16; ++j) d[j >> 2][j & 3] = *(const unsigned*)(src + (size_t)j * N);
      } };
    auto publish = [&](int n) {
      asm volatile("" ::: "memory"); __builtin_amdgcn_s_barrier(); asm volatile("" ::: "memory");
      if (wid == 0 && lane < NR) {
        unsigned* f = (unsigned*)(P.ws[lane] + CTL_OFF + CTL_FLAGS + (size_t)P.rank * 128) + cj;
        __hip_atomic_store(f, call * 32u + (unsigned)n, __ATOMIC_RELAXED, __HIP_MEMORY_SCOPE_SYSTEM); } };
    bool drained = false;
    int g = cj; load_unit(g);
    for (int it = 0; g < NUNIT; ++it, g += NCOMM) {
      const UD u = decode(g);
      const unsigned sbase_ = (unsigned)(P.rank * SRC_STRIDE) + (unsigned)(u.kq * HTB);
      u32x4 v[4]; unsigned off[4];
      if (u.isx) {
#pragma unroll
        for (int i = 0; i < 4; ++i) { const int p = tid + 512 * i, hb = p >> 10; v[i] = d[i];
          off[i] = sbase_ + (unsigned)((u.pan * 2 + hb) * HBSTRIDE) + (unsigned)((p & 1023) * 16); }
      } else {
        const int wc = wid >> 1, h = wid & 1;
#pragma unroll
        for (int q4 = 0; q4 < 4; ++q4) {
          const unsigned a0 = d[q4][0], a1 = d[q4][1], a2 = d[q4][2], a3 = d[q4][3];
          const unsigned t0 = perm(a1, a0, 0x05010400u), t1 = perm(a1, a0, 0x07030602u);
          const unsigned t2 = perm(a3, a2, 0x05010400u), t3 = perm(a3, a2, 0x07030602u);
          v[0][q4] = perm(t2, t0, 0x05040100u); v[1][q4] = perm(t2, t0, 0x07060302u);
          v[2][q4] = perm(t3, t1, 0x05040100u); v[3][q4] = perm(t3, t1, 0x07060302u);
        }
#pragma unroll
        for (int c = 0; c < 4; ++c) { const int hb = c >> 1, n = c & 1;
          off[c] = sbase_ + (unsigned)XS_BYTES + (unsigned)((u.pan * 2 + hb) * HBSTRIDE) + (unsigned)((wc * 2 + n) * 2048 + h * 1024 + lane * 16); }
      }
      if (g + NCOMM < NUNIT) load_unit(g + NCOMM);
#pragma unroll
      for (int hf = 0; hf < 2; ++hf) {
#pragma unroll
        for (int q = 0; q < NR; ++q) {
          const int pd = (P.rank + 1 + q + rot) & (NR - 1);
          const __amdgpu_buffer_rsrc_t rsq = __builtin_amdgcn_make_buffer_rsrc(P.ws[pd] + bufoff, 0, (unsigned)BUF_BYTES, 0x00020000);
#pragma unroll
          for (int i = 0; i < 2; ++i) __builtin_amdgcn_raw_buffer_store_b128(v[hf * 2 + i], rsq, off[hf * 2 + i], 0, COMM_AUX);
          if (hf == COMM_HF && q == 6 && it > 0 && !drained) { asm volatile("s_waitcnt vmcnt(%0)" :: "n"(16 * COMM_HF + 14) : "memory"); publish(it); STAMP(it); }
        }
      }
      drained = false;
      if (g >= DRAIN_BEG && g < DRAIN_END) { asm volatile("s_waitcnt vmcnt(0)" ::: "memory"); publish(it + 1); drained = true; }
    }
    asm volatile("s_waitcnt vmcnt(0)" ::: "memory");
    publish(NUNIT / NCOMM);
    STAMP(40);
  }

  const int wr = wid >> 2, wc = wid & 3, fr = lane & 15, fq = lane >> 4;
  const int gi = blockIdx.x - NCOMM, xg = gi & 7, slot = gi >> 3;
  const char* const As = myws + bufoff; const char* const Bs = As + XS_BYTES;
  const unsigned voff = tid * 16;
  const unsigned ldsw = (unsigned)wid * 1024u;
  const int aoff = wr * 4 * 2048 + lane * 16, boff = wc * 2 * 2048 + lane * 16;
  const unsigned ovoff = (unsigned)((wr * 64 + fq * 4) * N + wc * 64 + fr * 4) * 4u;
  int lvl = 0;
  bool bad = false;
#define SA(b, h) (((b) * 2 + (h)) * HTB)
#define SB(b, h) ((4 + (b) * 2 + (h)) * HTB)
#define STAGE(bufoff_, gbase) do { _Pragma("unroll") for (int _i = 0; _i < 2; ++_i) \
        __builtin_amdgcn_global_load_lds((const unsigned*)((const char*)(gbase) + voff + _i * 8192), (LAS unsigned*)(lds + (bufoff_) + ldsw + _i * 8192), 16, 0, 0); } while (0)
#define LDA(b, h, mp) do { _Pragma("unroll") for (int m = 0; m < 2; ++m) _Pragma("unroll") for (int k = 0; k < 2; ++k) At[m][k] = *(const LAS i32x4*)(lds + SA(b, h) + aoff + ((mp) * 2 + m) * 2048 + k * 1024); } while (0)
#define LDB(b, h) do { _Pragma("unroll") for (int n = 0; n < 2; ++n) _Pragma("unroll") for (int k = 0; k < 2; ++k) Bt[h][n][k] = *(const LAS i32x4*)(lds + SB(b, h) + boff + n * 2048 + k * 1024); } while (0)
#define CAT(x) __builtin_shufflevector(x[0], x[1], 0, 1, 2, 3, 4, 5, 6, 7)
#define MMA(ai, mp, ZERO) do { __builtin_amdgcn_s_setprio(1); _Pragma("unroll") for (int m = 0; m < 2; ++m) _Pragma("unroll") for (int h = 0; h < 2; ++h) _Pragma("unroll") for (int n = 0; n < 2; ++n) { \
        if (ZERO) asm volatile("v_mfma_f32_16x16x128_f8f6f4 %0, %1, %2, 0" : "=v"(acc[ai][(mp) * 2 + m][h][n]) : "v"(CAT(At[m])), "v"(CAT(Bt[h][n]))); \
        else asm volatile("v_mfma_f32_16x16x128_f8f6f4 %0, %1, %2, %0" : "+v"(acc[ai][(mp) * 2 + m][h][n]) : "v"(CAT(At[m])), "v"(CAT(Bt[h][n]))); } \
        __builtin_amdgcn_s_setprio(0); } while (0)
#define ST(ai, mp, s0, s1, sbase) do { _Pragma("unroll") for (int s = (s0); s < (s1); ++s) { const int m = (mp) * 2 + (s >> 2), j = s & 3; \
        f32x4 v = {acc[ai][m][0][0][j], acc[ai][m][0][1][j], acc[ai][m][1][0][j], acc[ai][m][1][1][j]}; v = v * scale; v = __builtin_elementwise_max(v, (f32x4){lo, lo, lo, lo}); \
        __builtin_amdgcn_raw_buffer_store_b128(__builtin_bit_cast(u32x4, v), orsrc, ovoff + ((sbase) + (unsigned)(((ai) * 128 + m * 16 + j) * N) * 4u), 0, ((ai) == 0) ? STAUX0 : STAUX1); } } while (0)
#define WAIT_V(n) asm volatile("s_waitcnt vmcnt(" #n ")" ::: "memory")
#define WAIT_L(n) asm volatile("s_waitcnt lgkmcnt(" #n ")" ::: "memory")
#define BAR __builtin_amdgcn_s_barrier()
#define SCHED __builtin_amdgcn_sched_barrier(0)
#define KOFF(t) ((size_t)((t) & 7) * SRC_STRIDE + (size_t)((t) >> 3) * HTB)
  auto level_from = [&](unsigned long long f0) -> int {
    const unsigned base = call * 32u; int B = 1 << 20;
#pragma unroll
    for (int k = 0; k < 2; ++k) {
      const unsigned v0 = (unsigned)(f0 >> (32 * k));
      const int n = (v0 > base && v0 - base < 32u) ? (int)(v0 - base) : 0;
      const int b = (n >= NUNIT / NCOMM) ? (1 << 20) : (lane & 7) * 2 + k + NCOMM * n;
      B = min(B, b);
    }
#pragma unroll
    for (int o = 32; o >= 1; o >>= 1) B = min(B, __shfl_xor(B, o));
    return level_of(__builtin_amdgcn_readfirstlane(B)); };
  const char* const fb = ctl + CTL_FLAGS + (lane >> 3) * 128 + (lane & 7) * 8;
  auto probe_level = [&]() -> int { return level_from(__hip_atomic_load((const unsigned long long*)fb, __ATOMIC_RELAXED, __HIP_MEMORY_SCOPE_SYSTEM)); };
#define WAIT_STAGE(s) do { if ((s) >= lvl) { \
      LAS unsigned* lw = (LAS unsigned*)(lds + STAGE_BYTES); \
      WACC_IN; \
      if (wid == 0) { \
        int l2; const unsigned long long t0 = __builtin_amdgcn_s_memrealtime(); bool dead = false; \
        for (;;) { \
          l2 = probe_level(); \
          if (l2 > (s)) break; \
          if (__hip_atomic_load(tmo, __ATOMIC_RELAXED, __HIP_MEMORY_SCOPE_AGENT) == call || __builtin_amdgcn_s_memrealtime() - t0 > DEADLINE_TICKS) { \
            if (lane == 0) __hip_atomic_store(tmo, call, __ATOMIC_RELAXED, __HIP_MEMORY_SCOPE_AGENT); \
            dead = true; l2 = NSTAGE; break; } \
          __builtin_amdgcn_s_sleep(POLL_SLEEP); \
        } \
        __builtin_amdgcn_fence(__ATOMIC_ACQUIRE, ""); \
        if (lane == 0) { lw[0] = (unsigned)l2; lw[1] = dead ? 1u : 0u; } \
        asm volatile("s_waitcnt vmcnt(0) lgkmcnt(0)" ::: "memory"); \
      } \
      asm volatile("" ::: "memory"); BAR; asm volatile("" ::: "memory"); \
      lvl = __builtin_amdgcn_readfirstlane(lw[0]); bad = bad || (__builtin_amdgcn_readfirstlane(lw[1]) != 0u); \
      asm volatile("s_waitcnt lgkmcnt(0)" ::: "memory"); \
      WACC_OUT; \
    } } while (0)

  float scale = __builtin_bit_cast(float, __builtin_amdgcn_readfirstlane(__builtin_bit_cast(int, P.sx[0] * P.sw[0])));
  float lo = 0.f;
  const __amdgpu_buffer_rsrc_t orsrc_real = __builtin_amdgcn_make_buffer_rsrc(P.out, 0, (unsigned)((size_t)M * N * 4), 0x00020000);
  {
    WAIT_STAGE(0);
    STAMP(1);
    if (bad) { scale = __builtin_nanf(""); lo = scale; }
    const int b = blockIdx.x, g8 = b & 7, j = b >> 3;
    const int pm = (g8 & 3) * 4 + (j & 3), pn = 60 + (g8 >> 2) * 2 + ((j >> 2) & 1), ai = (j >> 3) & 1, cb = (j >> 4) & 1;
    const int wr4 = wid >> 1, wc2 = wid & 1;
    const char* pA = As + (size_t)(pm * 2 + ai) * HBSTRIDE + voff;
    const char* pB0 = Bs + (size_t)(pn * 2) * HBSTRIDE + cb * 8192 + voff;
    const char* pB1 = pB0 + HBSTRIDE;
#define PSTAGE(t) do { const unsigned _sl = (unsigned)((t) & 3) * 32768u + ldsw; const size_t _ko = KOFF((t) & 31); \
      __builtin_amdgcn_global_load_lds((const unsigned*)(pA + _ko), (LAS unsigned*)(lds + _sl), 16, 0, 0); \
      __builtin_amdgcn_global_load_lds((const unsigned*)(pA + _ko + 8192), (LAS unsigned*)(lds + _sl + 8192), 16, 0, 0); \
      __builtin_amdgcn_global_load_lds((const unsigned*)(pB0 + _ko), (LAS unsigned*)(lds + _sl + 16384), 16, 0, 0); \
      __builtin_amdgcn_global_load_lds((const unsigned*)(pB1 + _ko), (LAS unsigned*)(lds + _sl + 24576), 16, 0, 0); } while (0)
    f32x4 pacc[2][2][2];
#pragma unroll
    for (int m = 0; m < 2; ++m)
#pragma unroll
      for (int h = 0; h < 2; ++h)
#pragma unroll
        for (int n = 0; n < 2; ++n) pacc[m][h][n] = (f32x4){0.f, 0.f, 0.f, 0.f};
    const int paoff = wr4 * 2 * 2048 + lane * 16, pboff = 16384 + wc2 * 2 * 2048 + lane * 16;
#define PLD(dstA, dstB, t) do { const unsigned _sl = (unsigned)((t) & 3) * 32768u; \
      _Pragma("unroll") for (int m = 0; m < 2; ++m) _Pragma("unroll") for (int k = 0; k < 2; ++k) dstA[m][k] = *(const LAS i32x4*)(lds + _sl + paoff + m * 2048 + k * 1024); \
      _Pragma("unroll") for (int h = 0; h < 2; ++h) _Pragma("unroll") for (int n = 0; n < 2; ++n) _Pragma("unroll") for (int k = 0; k < 2; ++k) dstB[h][n][k] = *(const LAS i32x4*)(lds + _sl + pboff + h * 8192 + n * 2048 + k * 1024); } while (0)
#define PMMA(srcA, srcB) do { _Pragma("unroll") for (int m = 0; m < 2; ++m) _Pragma("unroll") for (int h = 0; h < 2; ++h) _Pragma("unroll") for (int n = 0; n < 2; ++n) \
      asm volatile("v_mfma_f32_16x16x128_f8f6f4 %0, %1, %2, %0" : "+v"(pacc[m][h][n]) : "v"(CAT(srcA[m])), "v"(CAT(srcB[h][n]))); } while (0)
    i32x4 pa0[2][2], pb0[2][2][2], pa1[2][2], pb1[2][2][2];
    PSTAGE(0); PSTAGE(1); PSTAGE(2);
    WAIT_V(8); BAR; PLD(pa0, pb0, 0);
    for (int t = 0; t < NT; t += 2) {
      WAIT_V(4); WAIT_L(0); BAR;
      if (((t + 4) & 7) == 0 && t + 4 < NT) WAIT_STAGE((t + 4) >> 3);
      PSTAGE(t + 3); PLD(pa1, pb1, t + 1); SCHED;
      PMMA(pa0, pb0);
      WAIT_V(4); WAIT_L(0); BAR;
      PSTAGE(t + 4); PLD(pa0, pb0, t + 2); SCHED;
      PMMA(pa1, pb1);
    }
    asm volatile("s_nop 15\n\ts_nop 15" ::: "memory");
    if (bad) { scale = __builtin_nanf(""); lo = scale; }
    const unsigned pov = (unsigned)((pm * 256 + ai * 128 + wr4 * 32 + fq * 4) * N + pn * 256 + (cb * 2 + wc2) * 64 + fr * 4) * 4u;
#pragma unroll
    for (int m = 0; m < 2; ++m)
#pragma unroll
      for (int jj = 0; jj < 4; ++jj) {
        f32x4 v = {pacc[m][0][0][jj], pacc[m][0][1][jj], pacc[m][1][0][jj], pacc[m][1][1][jj]}; v = v * scale; v = __builtin_elementwise_max(v, (f32x4){lo, lo, lo, lo});
        __builtin_amdgcn_raw_buffer_store_b128(__builtin_bit_cast(u32x4, v), orsrc_real, pov + (unsigned)((m * 16 + jj) * N) * 4u, 0, 2); }
    WAIT_V(0); BAR;
    STAMP(22);
  }
  if (blockIdx.x >= NCOMM) {
  const int r_begin = 0, r_end = 4;
  Unit cur, nxt; int ui = r_begin;
  tile_of(xg, slot, ui, cur);
  f32x4 acc[2][4][2][2];
  i32x4 At[2][2], Bt[2][2][2];
  const size_t hstep = HBSTRIDE, tstep = 2 * HBSTRIDE;
  const char* cA = As + (size_t)cur.pm * tstep; const char* cB = Bs + (size_t)cur.pn * tstep;
  __amdgpu_buffer_rsrc_t orsrc = __builtin_amdgcn_make_buffer_rsrc(P.out, 0, 0u, 0x00020000);
  WAIT_STAGE(stage_of(r_begin, 0));
  STAMP(23);
  STAGE(SB(0, 0), cB); STAGE(SB(0, 1), cB + hstep); STAGE(SA(0, 0), cA); STAGE(SA(0, 1), cA + hstep);
  STAGE(SB(1, 0), cB + KOFF(1)); STAGE(SB(1, 1), cB + hstep + KOFF(1)); STAGE(SA(1, 0), cA + KOFF(1));
  if (wr == 1) BAR;
  WAIT_V(6); BAR; BAR;
  unsigned psb = 0;
#pragma unroll
  for (int a = 0; a < 2; ++a)
#pragma unroll
    for (int m = 0; m < 4; ++m)
#pragma unroll
      for (int h = 0; h < 2; ++h)
#pragma unroll
        for (int n = 0; n < 2; ++n) acc[a][m][h][n] = (f32x4){0.f, 0.f, 0.f, 0.f};
#define ITER(FIRST, LAST, a1, a2, b2, a3, b3) do { \
        LDB(0, 0); SCHED; LDB(0, 1); LDA(0, 0, 0); \
      if (FIRST) { ST(0, 1, 4, 8, psb); } \
      WAIT_L(8); BAR; WAIT_L(0); MMA(0, 0, FIRST); BAR; SCHED; \
        LDA(0, 0, 1); STAGE(SB(0, 0), b2); STAGE(SA(1, 1), (a1) + hstep); \
      if (FIRST) { ST(1, 0, 4, 8, psb); WAIT_V(24); } else WAIT_V(8); \
      BAR; WAIT_L(0); MMA(0, 1, FIRST); BAR; SCHED; \
        LDA(0, 1, 0); STAGE(SB(0, 1), (b2) + hstep); \
      if (FIRST) { ST(1, 1, 0, 4, psb); } \
      BAR; WAIT_L(0); MMA(1, 0, FIRST); BAR; SCHED; \
        LDA(0, 1, 1); STAGE(SA(0, 0), a2); \
      if (FIRST) { ST(1, 1, 4, 8, psb); WAIT_V(24); } else WAIT_V(8); \
      BAR; WAIT_L(0); MMA(1, 1, FIRST); BAR; SCHED; \
        LDB(1, 0); SCHED; LDB(1, 1); LDA(1, 0, 0); \
      WAIT_L(8); BAR; WAIT_L(0); MMA(0, 0, 0); BAR; SCHED; \
        LDA(1, 0, 1); STAGE(SB(1, 0), b3); STAGE(SA(0, 1), (a2) + hstep); \
      if (LAST) { ST(0, 0, 0, 4, sb); WAIT_V(12); } else if (FIRST) WAIT_V(16); else WAIT_V(8); \
      BAR; WAIT_L(0); MMA(0, 1, 0); BAR; SCHED; \
        LDA(1, 1, 0); STAGE(SB(1, 1), (b3) + hstep); \
      if (LAST) { ST(0, 0, 4, 8, sb); } \
      BAR; WAIT_L(0); MMA(1, 0, 0); BAR; SCHED; \
        LDA(1, 1, 1); STAGE(SA(1, 0), a3); \
      if (LAST) { ST(0, 1, 0, 4, sb); ST(1, 0, 0, 4, sb); WAIT_V(24); } else WAIT_V(8); \
      BAR; WAIT_L(0); MMA(1, 1, 0); BAR; SCHED; \
    } while (0)
  for (;;) {
    const bool has_next = (ui + 1 < r_end) && tile_of(xg, slot, ui + 1, nxt);
    const char* nA = has_next ? As + (size_t)nxt.pm * tstep : cA; const char* nB = has_next ? Bs + (size_t)nxt.pn * tstep : cB;
    const unsigned sb = (unsigned)(cur.pm * BM * N + cur.pn * BM) * 4u;
    ITER(1, 0, cA + KOFF(1), cA + KOFF(2), cB + KOFF(2), cA + KOFF(3), cB + KOFF(3));
    orsrc = orsrc_real;
    for (int t = 2; t < NT - 2; t += 2) {
      if (((t + 2) & 7) == 0) { WAIT_STAGE(stage_of(ui, (t + 2) >> 3)); }
      ITER(0, 0, cA + KOFF(t + 1), cA + KOFF(t + 2), cB + KOFF(t + 2), cA + KOFF(t + 3), cB + KOFF(t + 3));
    }
    STAMP(2 + ui * 2);
    if (has_next) WAIT_STAGE(stage_of(ui + 1, 0));
    if (bad) { scale = __builtin_nanf(""); lo = scale; }
    ITER(0, 1, cA + KOFF(NT - 1), nA, nB, nA + KOFF(1), nB + KOFF(1));
    psb = sb;
    if (!has_next) break;
    cur = nxt; cA = nA; cB = nB; ++ui;
  }
  ST(0, 1, 4, 8, psb); ST(1, 0, 4, 8, psb); ST(1, 1, 0, 8, psb);
  if (wr == 0) BAR;
  BAR;
  STAMP(20);
  }
  STAMP(63);
}
}

extern "C" void kernel_launch(int rank, int nranks, ncclComm_t comm,
                              hipStream_t stream,
                              void* const* d_in, const int* in_sizes, int n_in,
                              void* d_out, int out_size,
                              const int* in_dtypes, int out_dtype,
                              void* const* d_out_all,
                              void* const* d_ws_all, size_t ws_bytes) {
  if (nranks != NR || n_in < 4 || ws_bytes < CTL_OFF + CTL_BYTES || (unsigned)rank >= (unsigned)NR) { fprintf(stderr, "unsupported configuration\n"); return; }
  static bool init = false;
  if (!init) {
    HIP_CHECK(hipFuncSetAttribute((const void*)fused, hipFuncAttributeMaxDynamicSharedMemorySize, LDS_BYTES));
    init = true;
  }
  Params P;
  P.x = (const unsigned char*)d_in[0]; P.w = (const unsigned char*)d_in[1];
  P.sx = (const float*)d_in[2]; P.sw = (const float*)d_in[3];
  P.out = (float*)d_out; P.rank = rank;
  for (int p = 0; p < NR; ++p) P.ws[p] = (char*)d_ws_all[p];
  hipLaunchKernelGGL(fused, dim3(GRID), dim3(512), LDS_BYTES, stream, P);
}
```

```cpp
#include <hip/hip_runtime.h>
#include <rccl/rccl.h>
#include <cstdio>

#define HIP_CHECK(expr) do { hipError_t _e = (expr); if (_e != hipSuccess) { \
    fprintf(stderr, "HIP error %s at %s:%d\n", hipGetErrorString(_e), __FILE__, __LINE__); \
    return; } } while (0)

namespace {
#define LAS __attribute__((address_space(3)))
typedef float f32x4 __attribute__((ext_vector_type(4)));
typedef int i32x4 __attribute__((ext_vector_type(4)));
typedef unsigned u32x4 __attribute__((ext_vector_type(4)));
typedef unsigned u32x2 __attribute__((ext_vector_type(2)));

constexpr int NR = 8;
constexpr int M = 4096, N = 16384, K = 4096, KS = K / NR;
constexpr int BM = 256, HTB = 16384, STAGE_BYTES = 8 * HTB, NT = K / 128;
constexpr int LDS_BYTES = STAGE_BYTES + 256;
constexpr size_t HBSTRIDE = (size_t)4 * HTB;
constexpr size_t XS_BYTES = (size_t)M * KS;
constexpr size_t SRC_STRIDE = (size_t)(M + N) * KS;
constexpr size_t BUF_BYTES = NR * SRC_STRIDE;
constexpr size_t CTL_OFF = 2 * BUF_BYTES;
constexpr int NCOMM = 16;
constexpr int GRID = 256;
constexpr int NSTAGE = 20;
constexpr int NUNIT = 320;
constexpr size_t CTL_FLAGS = 0;
constexpr size_t CTL_CNT = 64 * 1024;
constexpr size_t CTL_TMO = 68 * 1024;
constexpr size_t CTL_BYTES = 72 * 1024;
constexpr int DRAIN_BEG = 124, DRAIN_END = 140;
#define COMM_AUX 17
#define STAUX0 2
#define STAUX1 0
#define POLL_SLEEP 8
constexpr unsigned long long DEADLINE_TICKS = 40000000ull;

struct Params {
  const unsigned char* x;
  const unsigned char* w;
  const float* sx; const float* sw;
  float* out;
  char* ws[NR];
  int rank;
};

__device__ __forceinline__ int level_of(int B) { return B < 80 ? B / 20 : (B >= NUNIT ? NSTAGE : 4 + (B - 80) / 15); }
__device__ __forceinline__ int stage_of(int r, int kq) { return 4 + 4 * r + kq; }
struct Unit { int pm, pn; };
__device__ __forceinline__ bool tile_of(int g, int j, int r, Unit& u) {
  const int band = g & 3, h = g >> 2;
  if (r < 4) { const int i = 30 * (2 * r + h) + j; u.pm = band * 4 + (i & 3); u.pn = i >> 2; return true; }
  return false;
}

__device__ __forceinline__ unsigned perm(unsigned a, unsigned b, unsigned sel) { return __builtin_amdgcn_perm(a, b, sel); }

__global__ __launch_bounds__(512, 2) void fused(Params P) {
  extern __shared__ __attribute__((aligned(16))) unsigned char shm[];
  LAS unsigned char* lds = (LAS unsigned char*)shm;
  const int tid = threadIdx.x, wid = __builtin_amdgcn_readfirstlane(tid >> 6), lane = tid & 63;
  char* const myws = P.ws[P.rank];
  char* const ctl = myws + CTL_OFF;
  unsigned call;
  {
    unsigned* cnt = (unsigned*)(ctl + CTL_CNT) + blockIdx.x;
    LAS unsigned* l = (LAS unsigned*)(lds + STAGE_BYTES);
    if (tid == 0) { unsigned c = __hip_atomic_load(cnt, __ATOMIC_RELAXED, __HIP_MEMORY_SCOPE_AGENT) + 1u; __hip_atomic_store(cnt, c, __ATOMIC_RELAXED, __HIP_MEMORY_SCOPE_AGENT); l[0] = c; }
    __syncthreads();
    call = __builtin_amdgcn_readfirstlane(l[0]);
    __syncthreads();
  }
  const size_t bufoff = (call & 1u) ? BUF_BYTES : 0;
  unsigned* const tmo = (unsigned*)(ctl + CTL_TMO);

  if (blockIdx.x < NCOMM) {
    const int cj = blockIdx.x;
    const int fr = lane & 15, fq = lane >> 4;
    u32x4 d[4];
    struct UD { int kq, isx, pan; };
    auto decode = [](int g) -> UD {
      UD u;
      const int a = g / 20, ai = g - a * 20;
      const int b = (g - 80) / 15, bi = (g - 80) - b * 15;
      const bool r0 = g < 80;
      u.kq = r0 ? a : (b & 3);
      u.isx = r0 && ai < 16;
      u.pan = r0 ? (ai < 16 ? ai : 44 + ai) : 15 * (b >> 2) + bi;
      return u; };
    auto load_unit = [&](int g) {
      const UD u = decode(g); const int kq = u.kq;
      if (u.isx) {
#pragma unroll
        for (int i = 0; i < 4; ++i) { const int p = tid + 512 * i, hb = p >> 10, gg = (p >> 7) & 7, h = (p >> 6) & 1;
          const int row = u.pan * 256 + hb * 128 + gg * 16 + fr;
          d[i] = *(const u32x4*)(P.x + (size_t)row * KS + kq * 128 + h * 64 + fq * 16); }
      } else {
        const int wc = wid >> 1, h = wid & 1;
        const unsigned char* src = P.w + (size_t)(kq * 128 + h * 64 + fq * 16) * N + u.pan * 256 + wc * 64 + fr * 4;
#pragma unroll
        for (int j = 0; j < 16; ++j) d[j >> 2][j & 3] = *(const unsigned*)(src + (size_t)j * N);
      } };
    auto publish = [&](int n) {
      asm volatile("" ::: "memory"); __builtin_amdgcn_s_barrier(); asm volatile("" ::: "memory");
      if (wid == 0 && lane < NR) {
        unsigned* f = (unsigned*)(P.ws[lane] + CTL_OFF + CTL_FLAGS + (size_t)P.rank * 128) + cj;
        __hip_atomic_store(f, call * 32u + (unsigned)n, __ATOMIC_RELAXED, __HIP_MEMORY_SCOPE_SYSTEM); } };
    bool drained = false;
    int g = cj; load_unit(g);
    for (int it = 0; g < NUNIT; ++it, g += NCOMM) {
      const UD u = decode(g);
      const unsigned sbase_ = (unsigned)(P.rank * SRC_STRIDE) + (unsigned)(u.kq * HTB);
      u32x4 v[4]; unsigned off[4];
      if (u.isx) {
#pragma unroll
        for (int i = 0; i < 4; ++i) { const int p = tid + 512 * i, hb = p >> 10; v[i] = d[i];
          off[i] = sbase_ + (unsigned)((u.pan * 2 + hb) * HBSTRIDE) + (unsigned)((p & 1023) * 16); }
      } else {
        const int wc = wid >> 1, h = wid & 1;
#pragma unroll
        for (int q4 = 0; q4 < 4; ++q4) {
          const unsigned a0 = d[q4][0], a1 = d[q4][1], a2 = d[q4][2], a3 = d[q4][3];
          const unsigned t0 = perm(a1, a0, 0x05010400u), t1 = perm(a1, a0, 0x07030602u);
          const unsigned t2 = perm(a3, a2, 0x05010400u), t3 = perm(a3, a2, 0x07030602u);
          v[0][q4] = perm(t2, t0, 0x05040100u); v[1][q4] = perm(t2, t0, 0x07060302u);
          v[2][q4] = perm(t3, t1, 0x05040100u); v[3][q4] = perm(t3, t1, 0x07060302u);
        }
#pragma unroll
        for (int c = 0; c < 4; ++c) { const int hb = c >> 1, n = c & 1;
          off[c] = sbase_ + (unsigned)XS_BYTES + (unsigned)((u.pan * 2 + hb) * HBSTRIDE) + (unsigned)((wc * 2 + n) * 2048 + h * 1024 + lane * 16); }
      }
      if (g + NCOMM < NUNIT) load_unit(g + NCOMM);
#pragma unroll
      for (int hf = 0; hf < 2; ++hf) {
#pragma unroll
        for (int q = 0; q < NR; ++q) {
          const int pd = (P.rank + 1 + q) & (NR - 1);
          const __amdgpu_buffer_rsrc_t rsq = __builtin_amdgcn_make_buffer_rsrc(P.ws[pd] + bufoff, 0, (unsigned)BUF_BYTES, 0x00020000);
#pragma unroll
          for (int i = 0; i < 2; ++i) __builtin_amdgcn_raw_buffer_store_b128(v[hf * 2 + i], rsq, off[hf * 2 + i], 0, COMM_AUX);
          if (hf == 1 && q == 6 && it > 0 && !drained) { asm volatile("s_waitcnt vmcnt(30)" ::: "memory"); publish(it); }
        }
      }
      drained = false;
      if (g >= DRAIN_BEG && g < DRAIN_END) { asm volatile("s_waitcnt vmcnt(0)" ::: "memory"); publish(it + 1); drained = true; }
    }
    asm volatile("s_waitcnt vmcnt(0)" ::: "memory");
    publish(NUNIT / NCOMM);
  }

  const int wr = wid >> 2, wc = wid & 3, fr = lane & 15, fq = lane >> 4;
  const int gi = blockIdx.x - NCOMM, xg = gi & 7, slot = gi >> 3;
  const char* const As = myws + bufoff; const char* const Bs = As + XS_BYTES;
  const unsigned voff = tid * 16;
  const unsigned ldsw = (unsigned)wid * 1024u;
  const int aoff = wr * 4 * 2048 + lane * 16, boff = wc * 2 * 2048 + lane * 16;
  const unsigned ovoff = (unsigned)((wr * 64 + fq * 4) * N + wc * 64 + fr * 4) * 4u;
  int lvl = 0;
  bool bad = false;
#define SA(b, h) (((b) * 2 + (h)) * HTB)
#define SB(b, h) ((4 + (b) * 2 + (h)) * HTB)
#define STAGE(bufoff_, gbase) do { _Pragma("unroll") for (int _i = 0; _i < 2; ++_i) \
        __builtin_amdgcn_global_load_lds((const unsigned*)((const char*)(gbase) + voff + _i * 8192), (LAS unsigned*)(lds + (bufoff_) + ldsw + _i * 8192), 16, 0, 0); } while (0)
#define LDA(b, h, mp) do { _Pragma("unroll") for (int m = 0; m < 2; ++m) _Pragma("unroll") for (int k = 0; k < 2; ++k) At[m][k] = *(const LAS i32x4*)(lds + SA(b, h) + aoff + ((mp) * 2 + m) * 2048 + k * 1024); } while (0)
#define LDB(b, h) do { _Pragma("unroll") for (int n = 0; n < 2; ++n) _Pragma("unroll") for (int k = 0; k < 2; ++k) Bt[h][n][k] = *(const LAS i32x4*)(lds + SB(b, h) + boff + n * 2048 + k * 1024); } while (0)
#define CAT(x) __builtin_shufflevector(x[0], x[1], 0, 1, 2, 3, 4, 5, 6, 7)
#define MMA(ai, mp, ZERO) do { __builtin_amdgcn_s_setprio(1); _Pragma("unroll") for (int m = 0; m < 2; ++m) _Pragma("unroll") for (int h = 0; h < 2; ++h) _Pragma("unroll") for (int n = 0; n < 2; ++n) { \
        if (ZERO) asm volatile("v_mfma_f32_16x16x128_f8f6f4 %0, %1, %2, 0" : "=v"(acc[ai][(mp) * 2 + m][h][n]) : "v"(CAT(At[m])), "v"(CAT(Bt[h][n]))); \
        else asm volatile("v_mfma_f32_16x16x128_f8f6f4 %0, %1, %2, %0" : "+v"(acc[ai][(mp) * 2 + m][h][n]) : "v"(CAT(At[m])), "v"(CAT(Bt[h][n]))); } \
        __builtin_amdgcn_s_setprio(0); } while (0)
#define ST(ai, mp, s0, s1, sbase) do { _Pragma("unroll") for (int s = (s0); s < (s1); ++s) { const int m = (mp) * 2 + (s >> 2), j = s & 3; \
        f32x4 v = {acc[ai][m][0][0][j], acc[ai][m][0][1][j], acc[ai][m][1][0][j], acc[ai][m][1][1][j]}; v = v * scale; v = __builtin_elementwise_max(v, (f32x4){lo, lo, lo, lo}); \
        __builtin_amdgcn_raw_buffer_store_b128(__builtin_bit_cast(u32x4, v), orsrc, ovoff + ((sbase) + (unsigned)(((ai) * 128 + m * 16 + j) * N) * 4u), 0, ((ai) == 0) ? STAUX0 : STAUX1); } } while (0)
#define WAIT_V(n) asm volatile("s_waitcnt vmcnt(" #n ")" ::: "memory")
#define WAIT_L(n) asm volatile("s_waitcnt lgkmcnt(" #n ")" ::: "memory")
#define BAR __builtin_amdgcn_s_barrier()
#define SCHED __builtin_amdgcn_sched_barrier(0)
#define KOFF(t) ((size_t)((t) & 7) * SRC_STRIDE + (size_t)((t) >> 3) * HTB)
  auto level_from = [&](unsigned long long f0) -> int {
    const unsigned base = call * 32u; int B = 1 << 20;
#pragma unroll
    for (int k = 0; k < 2; ++k) {
      const unsigned v0 = (unsigned)(f0 >> (32 * k));
      const int n = v0 > base ? (int)min(v0 - base, 31u) : 0;
      const int b = (n >= NUNIT / NCOMM) ? (1 << 20) : (lane & 7) * 2 + k + NCOMM * n;
      B = min(B, b);
    }
#pragma unroll
    for (int o = 32; o >= 1; o >>= 1) B = min(B, __shfl_xor(B, o));
    return level_of(__builtin_amdgcn_readfirstlane(B)); };
  const char* const fb = ctl + CTL_FLAGS + (lane >> 3) * 128 + (lane & 7) * 8;
  auto probe_level = [&]() -> int { return level_from(__hip_atomic_load((const unsigned long long*)fb, __ATOMIC_RELAXED, __HIP_MEMORY_SCOPE_SYSTEM)); };
#define WAIT_STAGE(s) do { if ((s) >= lvl) { \
      LAS unsigned* lw = (LAS unsigned*)(lds + STAGE_BYTES); \
      if (wid == 0) { \
        int l2; const unsigned long long t0 = __builtin_amdgcn_s_memrealtime(); bool dead = false; \
        for (;;) { \
          l2 = probe_level(); \
          if (l2 > (s)) break; \
          if (__hip_atomic_load(tmo, __ATOMIC_RELAXED, __HIP_MEMORY_SCOPE_AGENT) == call || __builtin_amdgcn_s_memrealtime() - t0 > DEADLINE_TICKS) { \
            if (lane == 0) __hip_atomic_store(tmo, call, __ATOMIC_RELAXED, __HIP_MEMORY_SCOPE_AGENT); \
            dead = true; l2 = NSTAGE; break; } \
          __builtin_amdgcn_s_sleep(POLL_SLEEP); \
        } \
        __builtin_amdgcn_fence(__ATOMIC_ACQUIRE, ""); \
        if (lane == 0) { lw[0] = (unsigned)l2; lw[1] = dead ? 1u : 0u; } \
        asm volatile("s_waitcnt vmcnt(0) lgkmcnt(0)" ::: "memory"); \
      } \
      asm volatile("" ::: "memory"); BAR; asm volatile("" ::: "memory"); \
      lvl = __builtin_amdgcn_readfirstlane(lw[0]); bad = bad || (__builtin_amdgcn_readfirstlane(lw[1]) != 0u); \
      asm volatile("s_waitcnt lgkmcnt(0)" ::: "memory"); \
    } } while (0)

  float scale = __builtin_bit_cast(float, __builtin_amdgcn_readfirstlane(__builtin_bit_cast(int, P.sx[0] * P.sw[0])));
  float lo = 0.f;
  const __amdgpu_buffer_rsrc_t orsrc_real = __builtin_amdgcn_make_buffer_rsrc(P.out, 0, (unsigned)((size_t)M * N * 4), 0x00020000);
  {
    WAIT_STAGE(0);
    if (bad) { scale = __builtin_nanf(""); lo = scale; }
    const int b = blockIdx.x, g8 = b & 7, j = b >> 3;
    const int pm = (g8 & 3) * 4 + (j & 3), pn = 60 + (g8 >> 2) * 2 + ((j >> 2) & 1), ai = (j >> 3) & 1, cb = (j >> 4) & 1;
    const int wr4 = wid >> 1, wc2 = wid & 1;
    const char* pA = As + (size_t)(pm * 2 + ai) * HBSTRIDE + voff;
    const char* pB0 = Bs + (size_t)(pn * 2) * HBSTRIDE + cb * 8192 + voff;
    const char* pB1 = pB0 + HBSTRIDE;
#define PSTAGE(t) do { const unsigned _sl = (unsigned)((t) & 3) * 32768u + ldsw; const size_t _ko = KOFF((t) & 31); \
      __builtin_amdgcn_global_load_lds((const unsigned*)(pA + _ko), (LAS unsigned*)(lds + _sl), 16, 0, 0); \
      __builtin_amdgcn_global_load_lds((const unsigned*)(pA + _ko + 8192), (LAS unsigned*)(lds + _sl + 8192), 16, 0, 0); \
      __builtin_amdgcn_global_load_lds((const unsigned*)(pB0 + _ko), (LAS unsigned*)(lds + _sl + 16384), 16, 0, 0); \
      __builtin_amdgcn_global_load_lds((const unsigned*)(pB1 + _ko), (LAS unsigned*)(lds + _sl + 24576), 16, 0, 0); } while (0)
    f32x4 pacc[2][2][2];
#pragma unroll
    for (int m = 0; m < 2; ++m)
#pragma unroll
      for (int h = 0; h < 2; ++h)
#pragma unroll
        for (int n = 0; n < 2; ++n) pacc[m][h][n] = (f32x4){0.f, 0.f, 0.f, 0.f};
    const int paoff = wr4 * 2 * 2048 + lane * 16, pboff = 16384 + wc2 * 2 * 2048 + lane * 16;
#define PLD(dstA, dstB, t) do { const unsigned _sl = (unsigned)((t) & 3) * 32768u; \
      _Pragma("unroll") for (int m = 0; m < 2; ++m) _Pragma("unroll") for (int k = 0; k < 2; ++k) dstA[m][k] = *(const LAS i32x4*)(lds + _sl + paoff + m * 2048 + k * 1024); \
      _Pragma("unroll") for (int h = 0; h < 2; ++h) _Pragma("unroll") for (int n = 0; n < 2; ++n) _Pragma("unroll") for (int k = 0; k < 2; ++k) dstB[h][n][k] = *(const LAS i32x4*)(lds + _sl + pboff + h * 8192 + n * 2048 + k * 1024); } while (0)
#define PMMA(srcA, srcB) do { _Pragma("unroll") for (int m = 0; m < 2; ++m) _Pragma("unroll") for (int h = 0; h < 2; ++h) _Pragma("unroll") for (int n = 0; n < 2; ++n) \
      asm volatile("v_mfma_f32_16x16x128_f8f6f4 %0, %1, %2, %0" : "+v"(pacc[m][h][n]) : "v"(CAT(srcA[m])), "v"(CAT(srcB[h][n]))); } while (0)
    i32x4 pa0[2][2], pb0[2][2][2], pa1[2][2], pb1[2][2][2];
    PSTAGE(0); PSTAGE(1); PSTAGE(2);
    WAIT_V(8); BAR; PLD(pa0, pb0, 0);
    for (int t = 0; t < NT; t += 2) {
      WAIT_V(4); WAIT_L(0); BAR;
      if (((t + 4) & 7) == 0 && t + 4 < NT) WAIT_STAGE((t + 4) >> 3);
      PSTAGE(t + 3); PLD(pa1, pb1, t + 1); SCHED;
      PMMA(pa0, pb0);
      WAIT_V(4); WAIT_L(0); BAR;
      PSTAGE(t + 4); PLD(pa0, pb0, t + 2); SCHED;
      PMMA(pa1, pb1);
    }
    asm volatile("s_nop 15\n\ts_nop 15" ::: "memory");
    if (bad) { scale = __builtin_nanf(""); lo = scale; }
    const unsigned pov = (unsigned)((pm * 256 + ai * 128 + wr4 * 32 + fq * 4) * N + pn * 256 + (cb * 2 + wc2) * 64 + fr * 4) * 4u;
#pragma unroll
    for (int m = 0; m < 2; ++m)
#pragma unroll
      for (int jj = 0; jj < 4; ++jj) {
        f32x4 v = {pacc[m][0][0][jj], pacc[m][0][1][jj], pacc[m][1][0][jj], pacc[m][1][1][jj]}; v = v * scale; v = __builtin_elementwise_max(v, (f32x4){lo, lo, lo, lo});
        __builtin_amdgcn_raw_buffer_store_b128(__builtin_bit_cast(u32x4, v), orsrc_real, pov + (unsigned)((m * 16 + jj) * N) * 4u, 0, 2); }
    WAIT_V(0); BAR;
  }
  if (blockIdx.x >= NCOMM) {
  const int r_begin = 0, r_end = 4;
  Unit cur, nxt; int ui = r_begin;
  tile_of(xg, slot, ui, cur);
  f32x4 acc[2][4][2][2];
  i32x4 At[2][2], Bt[2][2][2];
  const size_t hstep = HBSTRIDE, tstep = 2 * HBSTRIDE;
  const char* cA = As + (size_t)cur.pm * tstep; const char* cB = Bs + (size_t)cur.pn * tstep;
  __amdgpu_buffer_rsrc_t orsrc = __builtin_amdgcn_make_buffer_rsrc(P.out, 0, 0u, 0x00020000);
  WAIT_STAGE(stage_of(r_begin, 0));
  STAGE(SB(0, 0), cB); STAGE(SB(0, 1), cB + hstep); STAGE(SA(0, 0), cA); STAGE(SA(0, 1), cA + hstep);
  STAGE(SB(1, 0), cB + KOFF(1)); STAGE(SB(1, 1), cB + hstep + KOFF(1)); STAGE(SA(1, 0), cA + KOFF(1));
  if (wr == 1) BAR;
  WAIT_V(6); BAR; BAR;
  unsigned psb = 0;
#pragma unroll
  for (int a = 0; a < 2; ++a)
#pragma unroll
    for (int m = 0; m < 4; ++m)
#pragma unroll
      for (int h = 0; h < 2; ++h)
#pragma unroll
        for (int n = 0; n < 2; ++n) acc[a][m][h][n] = (f32x4){0.f, 0.f, 0.f, 0.f};
#define ITER(FIRST, LAST, a1, a2, b2, a3, b3) do { \
        LDB(0, 0); SCHED; LDB(0, 1); LDA(0, 0, 0); \
      if (FIRST) { ST(0, 1, 4, 8, psb); } \
      WAIT_L(8); BAR; WAIT_L(0); MMA(0, 0, FIRST); BAR; SCHED; \
        LDA(0, 0, 1); STAGE(SB(0, 0), b2); STAGE(SA(1, 1), (a1) + hstep); \
      if (FIRST) { ST(1, 0, 4, 8, psb); WAIT_V(24); } else WAIT_V(8); \
      BAR; WAIT_L(0); MMA(0, 1, FIRST); BAR; SCHED; \
        LDA(0, 1, 0); STAGE(SB(0, 1), (b2) + hstep); \
      if (FIRST) { ST(1, 1, 0, 4, psb); } \
      BAR; WAIT_L(0); MMA(1, 0, FIRST); BAR; SCHED; \
        LDA(0, 1, 1); STAGE(SA(0, 0), a2); \
      if (FIRST) { ST(1, 1, 4, 8, psb); WAIT_V(24); } else WAIT_V(8); \
      BAR; WAIT_L(0); MMA(1, 1, FIRST); BAR; SCHED; \
        LDB(1, 0); SCHED; LDB(1, 1); LDA(1, 0, 0); \
      WAIT_L(8); BAR; WAIT_L(0); MMA(0, 0, 0); BAR; SCHED; \
        LDA(1, 0, 1); STAGE(SB(1, 0), b3); STAGE(SA(0, 1), (a2) + hstep); \
      if (LAST) { ST(0, 0, 0, 4, sb); WAIT_V(12); } else if (FIRST) WAIT_V(16); else WAIT_V(8); \
      BAR; WAIT_L(0); MMA(0, 1, 0); BAR; SCHED; \
        LDA(1, 1, 0); STAGE(SB(1, 1), (b3) + hstep); \
      if (LAST) { ST(0, 0, 4, 8, sb); } \
      BAR; WAIT_L(0); MMA(1, 0, 0); BAR; SCHED; \
        LDA(1, 1, 1); STAGE(SA(1, 0), a3); \
      if (LAST) { ST(0, 1, 0, 4, sb); ST(1, 0, 0, 4, sb); WAIT_V(24); } else WAIT_V(8); \
      BAR; WAIT_L(0); MMA(1, 1, 0); BAR; SCHED; \
    } while (0)
  for (;;) {
    const bool has_next = (ui + 1 < r_end) && tile_of(xg, slot, ui + 1, nxt);
    const char* nA = has_next ? As + (size_t)nxt.pm * tstep : cA; const char* nB = has_next ? Bs + (size_t)nxt.pn * tstep : cB;
    const unsigned sb = (unsigned)(cur.pm * BM * N + cur.pn * BM) * 4u;
    ITER(1, 0, cA + KOFF(1), cA + KOFF(2), cB + KOFF(2), cA + KOFF(3), cB + KOFF(3));
    orsrc = orsrc_real;
    for (int t = 2; t < NT - 2; t += 2) {
      if (((t + 2) & 7) == 0) { WAIT_STAGE(stage_of(ui, (t + 2) >> 3)); }
      ITER(0, 0, cA + KOFF(t + 1), cA + KOFF(t + 2), cB + KOFF(t + 2), cA + KOFF(t + 3), cB + KOFF(t + 3));
    }
    if (has_next) WAIT_STAGE(stage_of(ui + 1, 0));
    if (bad) { scale = __builtin_nanf(""); lo = scale; }
    ITER(0, 1, cA + KOFF(NT - 1), nA, nB, nA + KOFF(1), nB + KOFF(1));
    psb = sb;
    if (!has_next) break;
    cur = nxt; cA = nA; cB = nB; ++ui;
  }
  ST(0, 1, 4, 8, psb); ST(1, 0, 4, 8, psb); ST(1, 1, 0, 8, psb);
  if (wr == 0) BAR;
  BAR;
  }
}
}

extern "C" void kernel_launch(int rank, int nranks, ncclComm_t comm,
                              hipStream_t stream,
                              void* const* d_in, const int* in_sizes, int n_in,
                              void* d_out, int out_size,
                              const int* in_dtypes, int out_dtype,
                              void* const* d_out_all,
                              void* const* d_ws_all, size_t ws_bytes) {
  if (nranks != NR || n_in < 4 || ws_bytes < CTL_OFF + CTL_BYTES || (unsigned)rank >= (unsigned)NR) { fprintf(stderr, "unsupported configuration\n"); return; }
  static bool init = false;
  if (!init) {
    HIP_CHECK(hipFuncSetAttribute((const void*)fused, hipFuncAttributeMaxDynamicSharedMemorySize, LDS_BYTES));
    init = true;
  }
  Params P;
  P.x = (const unsigned char*)d_in[0]; P.w = (const unsigned char*)d_in[1];
  P.sx = (const float*)d_in[2]; P.sw = (const float*)d_in[3];
  P.out = (float*)d_out; P.rank = rank;
  for (int p = 0; p < NR; ++p) P.ws[p] = (char*)d_ws_all[p];
  hipLaunchKernelGGL(fused, dim3(GRID), dim3(512), LDS_BYTES, stream, P);
}
```
